```python
import numpy as np

kernel_name = "gpu_a2a_gemm_m4096_k4096_n4096_e4m3e4m3_dqsilu"


ABSMAX_RTOL = 0.03

COMPUTE_DTYPE = "float32"
EPILOGUE = "dequant_silu"
IN_QUANT = None
OUT_QUANT = None
IN_FP8 = "e4m3e4m3"
IN_DTYPES = {"x": "e4m3", "w_mat": "e4m3",
             "scale_x": "float32", "scale_w": "float32"}
WORLD_SIZE = 8

def _snap_fp8(x, mant_bits, max_abs, min_normal):
    a = np.abs(x.astype(np.float64))
    _m, _e = np.frexp(a)
    q = np.where(a >= min_normal, np.ldexp(1.0, _e - 1 - mant_bits),
                 min_normal * 2.0 ** -mant_bits)
    snapped = np.minimum(np.round(a / q) * q, max_abs)
    return (np.sign(x) * snapped).astype(np.float32)

def setup_inputs():

    x = _snap_fp8(np.random.randn(4096, 4096).astype(np.float32), 3, 448.0, 2.0 ** -6)
    w_mat = _snap_fp8(np.random.randn(4096, 4096).astype(np.float32), 3, 448.0, 2.0 ** -6)
    scale_x = (0.02 * (1.0 + np.random.rand(1))).astype(np.float32)
    scale_w = (0.02 * (1.0 + np.random.rand(1))).astype(np.float32)
    return {"x": x, "w_mat": w_mat, "scale_x": scale_x, "scale_w": scale_w}

def reference(x, w_mat, scale_x, scale_w):
    acc = (x @ w_mat).astype(np.float32)
    y = (acc * (scale_x * scale_w)).astype(np.float32)
    return (y / (1.0 + np.exp(-np.clip(y, -60.0, 60.0)))).astype(np.float32)
```

```cpp
#include <hip/hip_runtime.h>
#include <rccl/rccl.h>
#include <cstdio>
#include <cstdint>

#define HIP_CHECK(expr) do { hipError_t _e = (expr); if (_e != hipSuccess) { \
    fprintf(stderr, "HIP error %s at %s:%d\n", hipGetErrorString(_e), __FILE__, __LINE__); \
    return; } } while (0)

typedef int v2i __attribute__((ext_vector_type(2)));
typedef int v4i __attribute__((ext_vector_type(4)));
typedef int v8i __attribute__((ext_vector_type(8)));
typedef float v4f __attribute__((ext_vector_type(4)));
typedef float v16f __attribute__((ext_vector_type(16)));

#define SLOT8 18
#define SLOT0 24
#define TBASE 60
#define JMAP(mt) (7 - (mt))
#define TSTEP1 4
#define TSTEP0 12
#define SPOL ""
#define ZSTEP 12
#define XPOL ""
#define PPOL "sc0 sc1"
#define APOL "sc1"
#define POLL_SLEEP 2

constexpr int NRANKS = 8;
constexpr int GRID = 256;
constexpr int LDS_STAGE = 131072;
constexpr int LDS_FLAGS = LDS_STAGE + 8192;
constexpr int LDS_JUNK = LDS_FLAGS + 64;
constexpr int LDS_BYTES = LDS_JUNK + 256;
constexpr size_t WS_CTR = 0;
constexpr size_t WS_DBG = 256 << 10;
constexpr size_t WS_XBUF = 1 << 20;
#define NBUFS 8
constexpr unsigned NBUF = NBUFS;
constexpr size_t XBUF_BYTES = 2 << 20;
constexpr size_t WS_NEED = WS_XBUF + NBUF * XBUF_BYTES;
#define DEADLINE 50000000ull
constexpr unsigned long long DEADLINE_TICKS = DEADLINE;

struct Args {
  const uint8_t* x; const uint8_t* W; const float* sx; const float* sw; float* out;
  uint8_t* myws;
  int rank, slow;
  uint8_t* peer[NRANKS];
};

#define WEND_TABLE {12, 20, 24, 28, 32, 36, 40, 44, 48, 52, 56, 60, 64, 64, 64, 64, 64}
constexpr int WEND[17] = WEND_TABLE;
#define A23GAP 3
constexpr int gap_lo(int s, int g) {
  int n = WEND[s + 1] - WEND[s];
  return WEND[s] + (n * g + 5) / 6;
}
constexpr int ops_before_step(int s) {
  int na = s < 2 ? 2 : (s + 2 < 16 ? s + 2 : 16);
  return 4 * na + 2 * WEND[s];
}
constexpr int posA(int s) {
  if (s < 2) return 16;
  if (s < 4) return ops_before_step(1) + 2 * (gap_lo(1, A23GAP) - WEND[1]) + 4 * (s - 1);
  return ops_before_step(s - 2) + 4;
}
constexpr int posW(int sl) {
  if (sl > 15) return 0;
  if (sl == 0) return 8;
  if (sl == 1) return 24;
  int last = 4 * sl + 3;
  if (last < WEND[0]) return 8 + 2 * (last + 1);
  int s = 0;
  while (WEND[s + 1] <= last) s++;
  int na = s == 0 ? 0 : s == 1 ? (last >= gap_lo(1, A23GAP) ? 8 : 0) : (s + 2 < 16 ? 4 : 0);
  return ops_before_step(s) + na + 2 * (last + 1 - WEND[s]);
}
constexpr int cmin(int a, int b) { return a < b ? a : b; }
constexpr int cmax(int a, int b) { return a > b ? a : b; }
constexpr int vm_normal(int s) { return cmin(63, ops_before_step(s) - cmax(posA(s), posW(s + 1))); }
constexpr int vm_mode(int s) { return s < 3 ? 0 : cmin(vm_normal(s), 2 * (WEND[s] - WEND[s - 1])); }
constexpr bool sched_ok() {
  if (WEND[0] < 8 || WEND[0] > 12 || WEND[16] != 64) return false;
  for (int s = 0; s < 16; s++) {
    if (WEND[s + 1] < WEND[s] || WEND[s + 1] - WEND[s] > 8) return false;
    if (WEND[s + 1] > 4 * (s + 5)) return false;
    if (s + 1 <= 15 && WEND[s] < 4 * (s + 2)) return false;
    if (vm_normal(s) < 0) return false;
  }
  return true;
}
static_assert(sched_ok(), "bad W schedule");

__device__ __forceinline__ v8i cat(v4i a, v4i b) { return __builtin_shufflevector(a, b, 0, 1, 2, 3, 4, 5, 6, 7); }
__device__ __forceinline__ v8i cat4(const v2i (&b)[4]) {
  v4i x = __builtin_shufflevector(b[0], b[1], 0, 1, 2, 3), y = __builtin_shufflevector(b[2], b[3], 0, 1, 2, 3);
  return cat(x, y);
}
__device__ __forceinline__ void loadA(v4i (&a)[4], unsigned voff, const uint8_t* base) {
  asm volatile("global_load_dwordx4 %0, %4, %5 " APOL "\n\t"
               "global_load_dwordx4 %1, %4, %5 offset:1024 " APOL "\n\t"
               "global_load_dwordx4 %2, %4, %5 offset:2048 " APOL "\n\t"
               "global_load_dwordx4 %3, %4, %5 offset:3072 " APOL
               : "=&v"(a[0]), "=&v"(a[1]), "=&v"(a[2]), "=&v"(a[3]) : "v"(voff), "s"(base) : "memory");
}
__device__ __forceinline__ void stageX(unsigned m0v, unsigned ve, unsigned vo, const uint8_t* base) {
  asm volatile("s_mov_b32 m0, %0\n\ts_nop 0\n\t"
               "global_load_lds_dwordx4 %1, %3 " XPOL "\n\tglobal_load_lds_dwordx4 %2, %3 offset:1024 " XPOL
               :: "s"(m0v), "v"(ve), "v"(vo), "s"(base) : "memory");
  asm volatile("s_mov_b32 m0, %0\n\ts_nop 0\n\t"
               "global_load_lds_dwordx4 %1, %3 " XPOL "\n\tglobal_load_lds_dwordx4 %2, %3 offset:1024 " XPOL
               :: "s"(m0v + 2048), "v"(ve), "v"(vo), "s"(base + 16 * 512) : "memory");
  asm volatile("s_mov_b32 m0, %0\n\ts_nop 0\n\t"
               "global_load_lds_dwordx4 %1, %3 " XPOL "\n\tglobal_load_lds_dwordx4 %2, %3 offset:1024 " XPOL
               :: "s"(m0v + 4096), "v"(ve), "v"(vo), "s"(base + 32 * 512) : "memory");
  asm volatile("s_mov_b32 m0, %0\n\ts_nop 0\n\t"
               "global_load_lds_dwordx4 %1, %3 " XPOL "\n\tglobal_load_lds_dwordx4 %2, %3 offset:1024 " XPOL
               :: "s"(m0v + 6144), "v"(ve), "v"(vo), "s"(base + 48 * 512) : "memory");
}
__device__ __forceinline__ void readX(v4i (&a)[4], unsigned a0, unsigned a1) {
  asm volatile("ds_read_b128 %0, %4\n\tds_read_b128 %1, %5\n\tds_read_b128 %2, %4 offset:4096\n\tds_read_b128 %3, %5 offset:4096"
               : "=&v"(a[0]), "=&v"(a[1]), "=&v"(a[2]), "=&v"(a[3]) : "v"(a0), "v"(a1) : "memory");
}
__device__ __forceinline__ void waitX(v4i (&a)[4], v4i (&b)[4]) {
  asm volatile("s_waitcnt lgkmcnt(0)" : "+v"(a[0]), "+v"(a[1]), "+v"(a[2]), "+v"(a[3]), "+v"(b[0]), "+v"(b[1]), "+v"(b[2]), "+v"(b[3]) :: "memory");
}
template <int OFF> __device__ __forceinline__ void dump16(const v16f& c) {
  asm volatile("ds_write_addtid_b32 %0 offset:%c16\n\tds_write_addtid_b32 %1 offset:%c17\n\tds_write_addtid_b32 %2 offset:%c18\n\tds_write_addtid_b32 %3 offset:%c19\n\t"
               "ds_write_addtid_b32 %4 offset:%c20\n\tds_write_addtid_b32 %5 offset:%c21\n\tds_write_addtid_b32 %6 offset:%c22\n\tds_write_addtid_b32 %7 offset:%c23\n\t"
               "ds_write_addtid_b32 %8 offset:%c24\n\tds_write_addtid_b32 %9 offset:%c25\n\tds_write_addtid_b32 %10 offset:%c26\n\tds_write_addtid_b32 %11 offset:%c27\n\t"
               "ds_write_addtid_b32 %12 offset:%c28\n\tds_write_addtid_b32 %13 offset:%c29\n\tds_write_addtid_b32 %14 offset:%c30\n\tds_write_addtid_b32 %15 offset:%c31"
               :: "v"(c[0]), "v"(c[1]), "v"(c[2]), "v"(c[3]), "v"(c[4]), "v"(c[5]), "v"(c[6]), "v"(c[7]),
                  "v"(c[8]), "v"(c[9]), "v"(c[10]), "v"(c[11]), "v"(c[12]), "v"(c[13]), "v"(c[14]), "v"(c[15]),
                  "i"(OFF), "i"(OFF + 256), "i"(OFF + 512), "i"(OFF + 768), "i"(OFF + 1024), "i"(OFF + 1280), "i"(OFF + 1536), "i"(OFF + 1792),
                  "i"(OFF + 2048), "i"(OFF + 2304), "i"(OFF + 2560), "i"(OFF + 2816), "i"(OFF + 3072), "i"(OFF + 3328), "i"(OFF + 3584), "i"(OFF + 3840) : "memory");
}
__device__ __forceinline__ void dmaW2(unsigned m0v, unsigned voff0, unsigned voff1, const uint8_t* base) {
  asm volatile("s_mov_b32 m0, %0\n\ts_nop 0\n\t"
               "global_load_lds_dwordx4 %1, %3\n\t"
               "global_load_lds_dwordx4 %2, %3 offset:1024"
               :: "s"(m0v), "v"(voff0), "v"(voff1), "s"(base) : "memory");
}
template <int OFF> __device__ __forceinline__ void readB(v2i (&b)[4], unsigned addr) {
  asm volatile("ds_read_b64_tr_b8 %0, %4 offset:%c5\n\t"
               "ds_read_b64_tr_b8 %1, %4 offset:%c6\n\t"
               "ds_read_b64_tr_b8 %2, %4 offset:%c7\n\t"
               "ds_read_b64_tr_b8 %3, %4 offset:%c8"
               : "=&v"(b[0]), "=&v"(b[1]), "=&v"(b[2]), "=&v"(b[3])
               : "v"(addr), "i"(OFF), "i"(OFF + 1024), "i"(OFF + 2048), "i"(OFF + 3072) : "memory");
}
template <int N> __device__ __forceinline__ void waitA(v4i (&a)[4]) {
  asm volatile("s_waitcnt vmcnt(%c4)" : "+v"(a[0]), "+v"(a[1]), "+v"(a[2]), "+v"(a[3]) : "i"(N) : "memory");
}
template <int N0, int N1> __device__ __forceinline__ void waitA2(v4i (&a)[4], v4i (&b)[4], int mode) {
  asm volatile("s_cmp_eq_u32 %8, 0\n\t"
               "s_cbranch_scc1 1f\n\t"
               "s_waitcnt vmcnt(%c10)\n"
               "1:\n\t"
               "s_waitcnt vmcnt(%c9)"
               : "+v"(a[0]), "+v"(a[1]), "+v"(a[2]), "+v"(a[3]), "+v"(b[0]), "+v"(b[1]), "+v"(b[2]), "+v"(b[3])
               : "s"(mode), "i"(N0), "i"(N1) : "memory", "scc");
}
template <int N> __device__ __forceinline__ void waitB(v2i (&b)[4]) {
  asm volatile("s_waitcnt lgkmcnt(%c4)" : "+v"(b[0]), "+v"(b[1]), "+v"(b[2]), "+v"(b[3]) : "i"(N) : "memory");
}
template <int N> __device__ __forceinline__ void waitB3(v2i (&b)[4], v2i (&c)[4], v2i (&d)[4]) {
  asm volatile("s_waitcnt lgkmcnt(%c12)" : "+v"(b[0]), "+v"(b[1]), "+v"(b[2]), "+v"(b[3]), "+v"(c[0]), "+v"(c[1]), "+v"(c[2]), "+v"(c[3]),
               "+v"(d[0]), "+v"(d[1]), "+v"(d[2]), "+v"(d[3]) : "i"(N) : "memory");
}
__device__ __forceinline__ unsigned umin3(unsigned a, unsigned b, unsigned c) { return min(min(a, b), c); }
__device__ __forceinline__ bool missing(const v4i (&a)[4]) {
  unsigned m0 = umin3(a[0].x, a[0].y, a[0].z), m1 = umin3(a[0].w, a[1].x, a[1].y);
  m0 = umin3(m0, a[1].z, a[1].w); m1 = umin3(m1, a[2].x, a[2].y);
  m0 = umin3(m0, a[2].z, a[2].w); m1 = umin3(m1, a[3].x, a[3].y);
  m0 = umin3(m0, a[3].z, a[3].w);
  return __builtin_amdgcn_ballot_w64(min(m0, m1) == 0u) != 0ull;
}

__device__ __forceinline__ int slice_of(int s, int cw, int rank) {
  if (s < 2) return rank * 8 + 2 * cw + s;
  int q = (s - 2) * 4 + cw;
  int j = (q * 37) >> 8;
  int i = q - j * 7;
  return ((rank + 1 + i) & 7) * 8 + j;
}

#define ST(i)

struct CW {
  v16f acc[2][4];
  v4i A[3][4];
  v2i B[4][4];
  unsigned baddr[4], wv0, wv1, av, lds0, trig_addr, trig_val;
  const uint8_t* Wn; const uint8_t* xb; const uint8_t* touch; unsigned junk;
  int cw, rank, mode, dead;
  unsigned long long t0;
  __device__ __forceinline__ const uint8_t* wbase(int sl) const { return Wn + (size_t)slice_of(sl, cw, rank) * (64 * 4096); }
  __device__ __forceinline__ const uint8_t* abase(int s) const { return xb + (size_t)slice_of(s, cw, rank) * (8 * 4096); }
  template <int P> __device__ __forceinline__ void pair() {
    dmaW2(lds0 + ((P >> 2) & 3) * 8192 + (P & 3) * 2048, wv0, wv1, wbase(P >> 2) + (P & 3) * (16 * 4096));
  }
  template <int S, int G> __device__ __forceinline__ void gap() {
    constexpr int lo = gap_lo(S, G), hi = gap_lo(S, G + 1);
    if constexpr (S == 1 && G == A23GAP) { loadA(A[2], av, abase(2)); loadA(A[0], av, abase(3)); }
    if constexpr (hi - lo >= 1) pair<lo>();
    if constexpr (hi - lo >= 2) pair<lo + 1>();
    static_assert(hi - lo <= 2, "");
    if constexpr (S == 15 && G == 0) asm volatile("s_mov_b32 m0, %0" :: "s"(lds0) : "memory");
    if constexpr (S == 15 && G >= 1) { dump16<(((G - 1) & 1) * 4 + ((G - 1) >> 1)) * 4096>(acc[(G - 1) & 1][(G - 1) >> 1]); __builtin_amdgcn_sched_barrier(0); }
  }
  template <int S, int I, int T> __device__ __forceinline__ void mf() {
    constexpr int U = S % 3;
    acc[I][T] = __builtin_amdgcn_mfma_scale_f32_32x32x64_f8f6f4(cat(A[U][2 * I], A[U][2 * I + 1]), cat4(B[T]), acc[I][T], 0, 0, 0, 0, 0, 0);
    __builtin_amdgcn_sched_barrier(0);
  }
  template <int S> __device__ __forceinline__ void step() {
    constexpr int U = S % 3, U1 = (S + 1) % 3, U2 = (S + 2) % 3;
    constexpr bool DOB = S < 15;
    constexpr int BOFF = ((S + 1) & 3) * 8192;
    if constexpr (S < 2) {
      waitA<vm_normal(S)>(A[U]);
    } else {
      waitA2<vm_normal(S), vm_mode(S)>(A[U], A[U1], mode);
      int look = mode;
      if (__builtin_expect(missing(A[U]), 0)) {
        do {
          if (dead) break;
          __builtin_amdgcn_s_sleep(POLL_SLEEP);
          loadA(A[U], av, abase(S));
          waitA2<0, 0>(A[U], A[U1], 0);
          if (__builtin_amdgcn_s_memrealtime() - t0 > DEADLINE_TICKS) dead = 1;
        } while (missing(A[U]));
        look = dead ? 0 : 1;
      }
      mode = 0;
      if constexpr (S < 15) {
        if (__builtin_expect(look, 0)) {
          if (missing(A[U1])) { loadA(A[U1], av, abase(S + 1)); mode = 1; }
        }
      }
    }
    ST(2 + S);
    if constexpr (S == TSTEP1 || S == TSTEP0) {
      if (touch) asm volatile("s_mov_b32 m0, %0\n\ts_nop 0\n\tglobal_load_lds_dword %1, %2"
                              :: "s"(junk), "v"(av << 8), "s"(touch + (S == TSTEP1 ? 256 * 4096 : 0)) : "memory");
    }
    if constexpr (S == ZSTEP) { if (cw == 0) asm volatile("ds_write_b32 %0, %1" :: "v"(trig_addr), "v"(trig_val) : "memory"); }
    if constexpr (S >= 2 && S + 2 < 16) loadA(A[U2], av, abase(S + 2));
    __builtin_amdgcn_sched_barrier(0);
    waitB<12>(B[0]);
    mf<S, 0, 0>(); mf<S, 1, 0>();
    if constexpr (DOB) readB<BOFF>(B[0], baddr[0]);
    waitB3<DOB ? 4 : 0>(B[1], B[2], B[3]);
    gap<S, 0>();
    mf<S, 0, 1>();
    gap<S, 1>();
    mf<S, 1, 1>();
    if constexpr (DOB) readB<BOFF>(B[1], baddr[1]);
    gap<S, 2>();
    mf<S, 0, 2>();
    gap<S, 3>();
    mf<S, 1, 2>();
    if constexpr (DOB) readB<BOFF>(B[2], baddr[2]);
    gap<S, 4>();
    mf<S, 0, 3>();
    gap<S, 5>();
    mf<S, 1, 3>();
    if constexpr (DOB) readB<BOFF>(B[3], baddr[3]);
    if constexpr (S == 1 && A23GAP == 6) { loadA(A[2], av, abase(2)); loadA(A[0], av, abase(3)); }
    __builtin_amdgcn_sched_barrier(0);
    if constexpr (S == 15) {
      dump16<(1 * 4 + 2) * 4096>(acc[1][2]);
      dump16<(0 * 4 + 3) * 4096>(acc[0][3]);
      asm volatile("s_nop 7" ::: "memory");
      dump16<(1 * 4 + 3) * 4096>(acc[1][3]);
    }
  }
};

__global__ __launch_bounds__(512) void a2a_gemm_kernel(Args a) {
  extern __shared__ __attribute__((aligned(16))) uint8_t smem[];
  const int b = blockIdx.x;
  const int lane = threadIdx.x & 63;
  const int w = __builtin_amdgcn_readfirstlane(threadIdx.x >> 6);
  const int xcd = b & 7, idx = b >> 3;
  const int ntile = ((xcd + 2 * (idx & 3)) & 7) + 8 * (idx & 3);
  const int mtile = idx >> 2;
  const int rank = a.rank;
  uint8_t* const myws = a.myws;
  unsigned* const ctr = (unsigned*)(myws + WS_CTR + (size_t)b * 128);
  unsigned call; float sxv, swv;
  asm volatile("" :: "s"(a.x), "s"(a.W), "s"(a.sx), "s"(a.sw), "s"(a.out), "s"(myws), "s"(a.slow), "s"(rank));
#define LOAD_SCALARS() asm volatile("s_load_dword %0, %3, 0x0\n\ts_load_dword %1, %4, 0x0\n\ts_load_dword %2, %5, 0x0\n\ts_waitcnt lgkmcnt(0)" \
               : "=&s"(call), "=&s"(sxv), "=&s"(swv) : "s"(ctr), "s"(a.sx), "s"(a.sw) : "memory")
  const unsigned xve = (unsigned)((lane >> 3) * 512 + (((lane & 7) ^ ((lane >> 4) & 3)) << 4));
  const unsigned xvo = (xve ^ 64u) + 8 * 512 - 1024;
  const unsigned xr0 = (unsigned)((lane & 31) * 128 + (((2 * (lane >> 5)) ^ ((lane >> 1) & 7)) << 4));

  if (w < 4) {
    __builtin_amdgcn_s_setprio(3);
    const unsigned long long t0 = __builtin_amdgcn_s_memrealtime();
    const int pj = JMAP(mtile), pm = ntile >> 2, pr = ntile & 3;
    const unsigned d0 = (unsigned)(rank + 1 + w) & 7u, d1 = (unsigned)(rank + 5 + w) & 7u;
    const unsigned stage = (unsigned)(uintptr_t)smem + LDS_STAGE + w * 2048;
    const uint8_t* src = a.x + (size_t)(pm * 64 + pr * 16) * 512 + pj * 64;
    const unsigned sv = (unsigned)((lane >> 2) * 512 + (lane & 3) * 16);
    asm volatile("s_mov_b32 m0, %0\n\ts_nop 0\n\tglobal_load_lds_dwordx4 %1, %2 " SPOL :: "s"(stage), "v"(sv), "s"(src + (size_t)d0 * (512 * 512)) : "memory");
    if (w < 3) asm volatile("s_mov_b32 m0, %0\n\ts_nop 0\n\tglobal_load_lds_dwordx4 %1, %2 " SPOL :: "s"(stage + 1024), "v"(sv), "s"(src + (size_t)d1 * (512 * 512)) : "memory");
    uint8_t* const ws0 = a.peer[w];
    uint8_t* const ws1 = a.peer[w + 4];
    LOAD_SCALARS();
    const size_t doff = WS_XBUF + (size_t)(call % NBUF) * XBUF_BYTES + (size_t)(rank * 64 + pj * 8 + pm) * 4096 + (pr >> 1) * 2048 + (pr & 1) * 256;
    const unsigned so = (unsigned)(((lane >> 4) & 1) * 1024 + (lane >> 5) * 512 + (lane & 15) * 16);
    const unsigned ra = stage + (lane & 15) * 64 + (lane >> 4) * 16;
    v4i v0, v1;
    asm volatile("s_waitcnt vmcnt(0)\n\tds_read_b128 %0, %2\n\tds_read_b128 %1, %2 offset:1024\n\ts_waitcnt lgkmcnt(0)"
                 : "=&v"(v0), "=&v"(v1) : "v"(ra) : "memory");
    v0.x = v0.x ? v0.x : 0x80; v0.y = v0.y ? v0.y : 0x80; v0.z = v0.z ? v0.z : 0x80; v0.w = v0.w ? v0.w : 0x80;
    v1.x = v1.x ? v1.x : 0x80; v1.y = v1.y ? v1.y : 0x80; v1.z = v1.z ? v1.z : 0x80; v1.w = v1.w ? v1.w : 0x80;
    const int slot = pj * 32 + ntile - SLOT0;
    if (slot > 0) {
      const unsigned long long due = t0 + TBASE + (((unsigned)slot * SLOT8) >> 3);
      while (__builtin_amdgcn_s_memrealtime() < due) __builtin_amdgcn_s_sleep(1);
    }
    asm volatile("global_store_dwordx4 %0, %1, %2 " PPOL "\n\ts_nop 1" :: "v"(so), "v"(v0), "s"(ws0 + doff) : "memory");
    if (w < 3) asm volatile("global_store_dwordx4 %0, %1, %2 " PPOL "\n\ts_nop 1" :: "v"(so), "v"(v1), "s"(ws1 + doff) : "memory");
    __builtin_amdgcn_s_setprio(0);
    if (w == 3) {
      const unsigned taddr = (unsigned)(uintptr_t)smem + LDS_FLAGS + 16;
      for (;;) {
        unsigned v;
        asm volatile("ds_read_b32 %0, %1\n\ts_waitcnt lgkmcnt(0)" : "=v"(v) : "v"(taddr) : "memory");
        if (__builtin_amdgcn_readfirstlane(v) == call + 0x5a5a0001u) break;
        if (__builtin_amdgcn_s_memrealtime() - t0 > 2000) break;
        __builtin_amdgcn_s_sleep(4);
      }
      uint8_t* z = myws + WS_XBUF + (size_t)((call + 2u) % NBUF) * XBUF_BYTES + (size_t)b * 1024;
      const v4i zero = {0, 0, 0, 0};
      const unsigned zo = lane * 16;
#pragma unroll
      for (int i = 1; i < 8; i++)
        asm volatile("global_store_dwordx4 %0, %1, %2 sc1\n\ts_nop 1" :: "v"(zo), "v"(zero), "s"(z + (size_t)((rank + i) & 7) * 262144) : "memory");
    }
  } else {
    CW c;
    c.cw = w - 4; c.rank = rank; c.mode = 0; c.dead = 0;
    c.t0 = __builtin_amdgcn_s_memrealtime();
    c.lds0 = (unsigned)(uintptr_t)smem + c.cw * 32768;
    c.wv0 = (unsigned)((lane >> 3) * 4096 + (((lane & 7) ^ (((lane >> 4) & 3) << 1)) << 4));
    c.wv1 = c.wv0 + 8 * 4096 - 1024;
    c.Wn = a.W + ntile * 128;
    c.av = lane * 16;
    {
      int g = lane >> 4, h = g >> 1, cg = g & 1, i = lane & 15, q = i >> 1, p = i & 1;
      int sw = ((q >> 1) & 3) << 1;
#pragma unroll
      for (int t = 0; t < 4; t++) c.baddr[t] = c.lds0 + (32 * h + q) * 128 + (((2 * t + cg) ^ sw) << 4) + 8 * p;
    }
#pragma unroll
    for (int i = 0; i < 2; i++)
#pragma unroll
      for (int t = 0; t < 4; t++) c.acc[i][t] = (v16f){};
    c.pair<0>(); c.pair<1>(); c.pair<2>(); c.pair<3>();
    const unsigned stage = c.lds0 + 3 * 8192;
    stageX(stage, xve, xvo, a.x + (size_t)(rank * 512 + mtile * 64) * 512 + c.cw * 128);
    c.pair<4>(); c.pair<5>(); c.pair<6>(); c.pair<7>();
    if constexpr (WEND[0] > 8) { c.pair<8>(); } if constexpr (WEND[0] > 9) { c.pair<9>(); }
    if constexpr (WEND[0] > 10) { c.pair<10>(); } if constexpr (WEND[0] > 11) { c.pair<11>(); }
    LOAD_SCALARS();
    {
      const int dlo = (a.slow - xcd) & 7;
      c.junk = (unsigned)(uintptr_t)smem + LDS_JUNK;
      c.touch = ((dlo & 1) == 0 && c.cw == 1) ? a.W + (a.slow + 8 * (dlo >> 1)) * 128 + (size_t)((idx >> 2) * 512 + (idx & 3) * 64) * 4096 : nullptr;
    }
    c.trig_addr = (unsigned)(uintptr_t)smem + LDS_FLAGS + 16; c.trig_val = call + 0x5a5a0001u;
    c.xb = myws + WS_XBUF + (size_t)(call % NBUF) * XBUF_BYTES + (size_t)mtile * 4096;
    asm volatile("s_waitcnt vmcnt(%c0)" :: "i"(ops_before_step(0) - posA(0)) : "memory");
    readX(c.A[0], stage + xr0, stage + (xr0 ^ 16u));
    readX(c.A[1], stage + (xr0 ^ 64u), stage + (xr0 ^ 80u));
    waitX(c.A[0], c.A[1]);
    readB<0>(c.B[0], c.baddr[0]); readB<0>(c.B[1], c.baddr[1]); readB<0>(c.B[2], c.baddr[2]); readB<0>(c.B[3], c.baddr[3]);
    c.step<0>(); c.step<1>(); c.step<2>(); c.step<3>(); c.step<4>(); c.step<5>(); c.step<6>(); c.step<7>();
    c.step<8>(); c.step<9>(); c.step<10>(); c.step<11>(); c.step<12>(); c.step<13>(); c.step<14>(); c.step<15>();
    if (lane == 0) ((int*)(smem + LDS_FLAGS))[c.cw] = c.dead;
  }
  asm volatile("s_waitcnt lgkmcnt(0)" ::: "memory");
  __builtin_amdgcn_s_barrier();
  asm volatile("" ::: "memory");
  {
    const v4i df = *(const v4i*)(smem + LDS_FLAGS);
    const float sc = (df.x | df.y | df.z | df.w) ? __builtin_nanf("") : sxv * swv;
    const float k1 = -sc * 1.4426950408889634f, lim = 60.0f * 1.4426950408889634f;
    const int n4 = lane & 7, t = ((lane >> 3) & 1) | ((lane >> 5) << 1), h = (lane >> 4) & 1;
    const int i = w >> 2, g = w & 3;
    const uint8_t* p = smem + ((i * 4 + t) * 16 + g * 4) * 256 + h * 128 + n4 * 16;
    float* o = a.out + (size_t)(mtile * 64 + i * 32 + g * 8 + h * 4) * 4096 + ntile * 128 + t * 32 + n4 * 4;
    v4f part[4][4];
#pragma unroll
    for (int e = 0; e < 4; e++)
#pragma unroll
      for (int k = 0; k < 4; k++) part[e][k] = *(const v4f*)(p + e * 256 + k * 32768);
#pragma unroll
    for (int e = 0; e < 4; e++) {
      v4f v = (part[e][0] + part[e][1]) + (part[e][2] + part[e][3]);
      v4f r;
#pragma unroll
      for (int k = 0; k < 4; k++) {
        float u = __builtin_amdgcn_fmed3f(v[k] * k1, -lim, lim);
        r[k] = (v[k] * sc) * __builtin_amdgcn_rcpf(1.0f + __builtin_amdgcn_exp2f(u));
      }
      asm volatile("global_store_dwordx4 %0, %1, off sc1\n\ts_nop 1" :: "v"(o + (size_t)e * 4096), "v"(r) : "memory");
    }
  }
  if (threadIdx.x == 0) __hip_atomic_store(ctr, call + 1u, __ATOMIC_RELAXED, __HIP_MEMORY_SCOPE_AGENT);
}

extern "C" void kernel_launch(int rank, int nranks, ncclComm_t comm, hipStream_t stream,
                              void* const* d_in, const int* in_sizes, int n_in,
                              void* d_out, int out_size, const int* in_dtypes, int out_dtype,
                              void* const* d_out_all, void* const* d_ws_all, size_t ws_bytes) {
  if (nranks != NRANKS || ws_bytes < WS_NEED || (unsigned)rank >= (unsigned)NRANKS) {
    fprintf(stderr, "unsupported configuration\n");
    return;
  }
  static bool init = false;
  if (!init) {
    HIP_CHECK(hipFuncSetAttribute((const void*)a2a_gemm_kernel, hipFuncAttributeMaxDynamicSharedMemorySize, LDS_BYTES));
    init = true;
  }
  Args a;
  a.x = (const uint8_t*)d_in[0]; a.W = (const uint8_t*)d_in[1];
  a.sx = (const float*)d_in[2]; a.sw = (const float*)d_in[3];
  a.out = (float*)d_out;
  a.myws = (uint8_t*)d_ws_all[rank];
  for (int p = 0; p < NRANKS; p++) a.peer[p] = (uint8_t*)d_ws_all[(rank + 1 + p) % NRANKS];
  a.rank = rank; a.slow = (int)((3u - (unsigned)((uintptr_t)d_in[1] >> 7)) & 7u);
  a2a_gemm_kernel<<<GRID, 512, LDS_BYTES, stream>>>(a);
}
```

```cpp
#include <hip/hip_runtime.h>
#include <rccl/rccl.h>
#include <cstdio>
#include <cstdint>

#define HIP_CHECK(expr) do { hipError_t _e = (expr); if (_e != hipSuccess) { \
    fprintf(stderr, "HIP error %s at %s:%d\n", hipGetErrorString(_e), __FILE__, __LINE__); \
    return; } } while (0)

typedef int v2i __attribute__((ext_vector_type(2)));
typedef int v4i __attribute__((ext_vector_type(4)));
typedef int v8i __attribute__((ext_vector_type(8)));
typedef float v4f __attribute__((ext_vector_type(4)));
typedef float v16f __attribute__((ext_vector_type(16)));

#define SLOT8 18
#define SLOT0 24
#define TBASE 60
#define JMAP(mt) (7 - (mt))
#define ZSTEP 13
#define POLL_SLEEP 2
#define GSLEEP 1
#define ROTM 1
#define ROTS 1
#define HU_TABLE {8, 8, 4, 4}

constexpr int NRANKS = 8;
constexpr int GRID = 256;
constexpr int LDS_STAGE = 131072;
constexpr int LDS_FLAGS = LDS_STAGE + 8192;
constexpr int LDS_JUNK = LDS_FLAGS + 128;
constexpr int LDS_BYTES = LDS_JUNK + 256;
constexpr size_t WS_CTR = 0;
constexpr size_t WS_DBG = 256 << 10;
constexpr size_t WS_XBUF = 1 << 20;
constexpr unsigned NBUF = 8;
constexpr size_t XBUF_BYTES = 2 << 20;
constexpr size_t WS_NEED = WS_XBUF + NBUF * XBUF_BYTES;
#define DEADLINE 50000000ull
constexpr unsigned long long DEADLINE_TICKS = DEADLINE;

struct Args {
  const uint8_t* x; const uint8_t* W; const float* sx; const float* sw; float* out;
  uint8_t* myws;
  int rank, slow;
  uint8_t* peer[NRANKS];
};

#define WEND_TABLE {12, 20, 24, 28, 32, 36, 40, 44, 48, 52, 56, 60, 64, 64, 64}
#define AST_TABLE {0, 0, 1, 1, 2, 3, 4, 5, 6, 7, 8, 9, 10, 11, 12, 12}
#define AGP_TABLE {0, 0, 3, 3, -1, -1, -1, -1, -1, -1, -1, -1, -1, -1, -1, -1}
constexpr int WEND[15] = WEND_TABLE;
constexpr int AST[16] = AST_TABLE;
constexpr int AGP[16] = AGP_TABLE;
constexpr int HU[4] = HU_TABLE;
constexpr int gap_lo(int s, int g) {
  int n = WEND[s + 1] - WEND[s];
  return WEND[s] + (n * g + 5) / 6;
}
constexpr int cmin(int a, int b) { return a < b ? a : b; }
constexpr int cmax(int a, int b) { return a > b ? a : b; }
struct Sched {
  int posA[16], posW[16], tot[15];
  constexpr Sched() : posA(), posW(), tot() {
    int n = 0;
    n += 8; posW[0] = n;
    n += 8; posA[0] = posA[1] = n;
    for (int p = 4; p < WEND[0]; p++) { n += 2; if ((p & 3) == 3) posW[p >> 2] = n; }
    for (int s = 0; s < 14; s++) {
      tot[s] = n;
      for (int g = -1; g <= 6; g++) {
        for (int a = 2; a < 16; a++) if (AST[a] == s && AGP[a] == g) { n += 4; posA[a] = n; }
        if (g >= 0 && g < 6) for (int p = gap_lo(s, g); p < gap_lo(s, g + 1); p++) { n += 2; if ((p & 3) == 3) posW[p >> 2] = n; }
      }
    }
    tot[14] = n;
  }
};
constexpr Sched SC;
constexpr int need_normal(int s) { return s == 14 ? SC.posA[14] : cmax(SC.posA[s], cmax(SC.posW[s + 1], s == 13 ? SC.posW[15] : 0)); }
constexpr int vm_normal(int s) { return cmin(63, SC.tot[s] - need_normal(s)); }
constexpr bool can_look(int s) { return s < 15 && AST[s + 1] < s; }
constexpr int vm_mode(int s) {
  return s < 3 ? 0 : cmin(63, SC.tot[s] - cmax(need_normal(s), cmax(SC.tot[s - 1], can_look(s) ? SC.posA[s + 1] : 0)));
}
constexpr bool sched_ok() {
  if (WEND[0] < 8 || WEND[0] > 12 || WEND[13] != 64) return false;
  for (int s = 0; s < 14; s++) {
    if (WEND[s + 1] < WEND[s] || WEND[s + 1] - WEND[s] > 8) return false;
    if (WEND[s + 1] > 4 * (s + 5)) return false;
    if (WEND[s] < 4 * (s + 2)) return false;
    if (vm_normal(s) < 0 || vm_mode(s) < 0) return false;
  }
  for (int a = 2; a < 16; a++) {
    if (AST[a] >= cmin(a, 14) || AST[a] < a - 3 || AST[a] < 0) return false;
    if (a > 2 && SC.posA[a] < SC.posA[a - 1]) return false;
  }
  return true;
}
static_assert(sched_ok(), "bad schedule");

__device__ __forceinline__ v8i cat(v4i a, v4i b) { return __builtin_shufflevector(a, b, 0, 1, 2, 3, 4, 5, 6, 7); }
__device__ __forceinline__ v8i cat4(const v2i (&b)[4]) {
  v4i x = __builtin_shufflevector(b[0], b[1], 0, 1, 2, 3), y = __builtin_shufflevector(b[2], b[3], 0, 1, 2, 3);
  return cat(x, y);
}
__device__ __forceinline__ void loadA(v4i (&a)[4], unsigned voff, const uint8_t* base) {
  asm volatile("global_load_dwordx4 %0, %4, %5 sc1\n\t"
               "global_load_dwordx4 %1, %4, %5 offset:1024 sc1\n\t"
               "global_load_dwordx4 %2, %4, %5 offset:2048 sc1\n\t"
               "global_load_dwordx4 %3, %4, %5 offset:3072 sc1"
               : "=&v"(a[0]), "=&v"(a[1]), "=&v"(a[2]), "=&v"(a[3]) : "v"(voff), "s"(base) : "memory");
}
__device__ __forceinline__ void dma2(unsigned m0v, unsigned voff0, unsigned voff1, const uint8_t* base) {
  asm volatile("s_mov_b32 m0, %0\n\ts_nop 0\n\t"
               "global_load_lds_dwordx4 %1, %3\n\t"
               "global_load_lds_dwordx4 %2, %3 offset:1024"
               :: "s"(m0v), "v"(voff0), "v"(voff1), "s"(base) : "memory");
}
__device__ __forceinline__ void readX(v4i (&a)[4], unsigned a0, unsigned a1) {
  asm volatile("ds_read_b128 %0, %4\n\tds_read_b128 %1, %5\n\tds_read_b128 %2, %4 offset:4096\n\tds_read_b128 %3, %5 offset:4096"
               : "=&v"(a[0]), "=&v"(a[1]), "=&v"(a[2]), "=&v"(a[3]) : "v"(a0), "v"(a1) : "memory");
}
__device__ __forceinline__ void waitX(v4i (&a)[4], v4i (&b)[4]) {
  asm volatile("s_waitcnt lgkmcnt(0)" : "+v"(a[0]), "+v"(a[1]), "+v"(a[2]), "+v"(a[3]), "+v"(b[0]), "+v"(b[1]), "+v"(b[2]), "+v"(b[3]) :: "memory");
}
template <int OFF> __device__ __forceinline__ void dump16(const v16f& c) {
  asm volatile("ds_write_addtid_b32 %0 offset:%c16\n\tds_write_addtid_b32 %1 offset:%c17\n\tds_write_addtid_b32 %2 offset:%c18\n\tds_write_addtid_b32 %3 offset:%c19\n\t"
               "ds_write_addtid_b32 %4 offset:%c20\n\tds_write_addtid_b32 %5 offset:%c21\n\tds_write_addtid_b32 %6 offset:%c22\n\tds_write_addtid_b32 %7 offset:%c23\n\t"
               "ds_write_addtid_b32 %8 offset:%c24\n\tds_write_addtid_b32 %9 offset:%c25\n\tds_write_addtid_b32 %10 offset:%c26\n\tds_write_addtid_b32 %11 offset:%c27\n\t"
               "ds_write_addtid_b32 %12 offset:%c28\n\tds_write_addtid_b32 %13 offset:%c29\n\tds_write_addtid_b32 %14 offset:%c30\n\tds_write_addtid_b32 %15 offset:%c31"
               :: "v"(c[0]), "v"(c[1]), "v"(c[2]), "v"(c[3]), "v"(c[4]), "v"(c[5]), "v"(c[6]), "v"(c[7]),
                  "v"(c[8]), "v"(c[9]), "v"(c[10]), "v"(c[11]), "v"(c[12]), "v"(c[13]), "v"(c[14]), "v"(c[15]),
                  "i"(OFF), "i"(OFF + 256), "i"(OFF + 512), "i"(OFF + 768), "i"(OFF + 1024), "i"(OFF + 1280), "i"(OFF + 1536), "i"(OFF + 1792),
                  "i"(OFF + 2048), "i"(OFF + 2304), "i"(OFF + 2560), "i"(OFF + 2816), "i"(OFF + 3072), "i"(OFF + 3328), "i"(OFF + 3584), "i"(OFF + 3840) : "memory");
}
template <int OFF> __device__ __forceinline__ void readB(v2i (&b)[4], unsigned addr) {
  asm volatile("ds_read_b64_tr_b8 %0, %4 offset:%c5\n\t"
               "ds_read_b64_tr_b8 %1, %4 offset:%c6\n\t"
               "ds_read_b64_tr_b8 %2, %4 offset:%c7\n\t"
               "ds_read_b64_tr_b8 %3, %4 offset:%c8"
               : "=&v"(b[0]), "=&v"(b[1]), "=&v"(b[2]), "=&v"(b[3])
               : "v"(addr), "i"(OFF), "i"(OFF + 1024), "i"(OFF + 2048), "i"(OFF + 3072) : "memory");
}
template <int N> __device__ __forceinline__ void waitA(v4i (&a)[4]) {
  asm volatile("s_waitcnt vmcnt(%c4)" : "+v"(a[0]), "+v"(a[1]), "+v"(a[2]), "+v"(a[3]) : "i"(N) : "memory");
}
template <int N> __device__ __forceinline__ void waitAA(v4i (&a)[4], v4i (&b)[4]) {
  asm volatile("s_waitcnt vmcnt(%c8)" : "+v"(a[0]), "+v"(a[1]), "+v"(a[2]), "+v"(a[3]), "+v"(b[0]), "+v"(b[1]), "+v"(b[2]), "+v"(b[3]) : "i"(N) : "memory");
}
template <int N0, int N1> __device__ __forceinline__ void waitA1m(v4i (&a)[4], int mode) {
  asm volatile("s_cmp_eq_u32 %4, 0\n\ts_cbranch_scc1 1f\n\ts_waitcnt vmcnt(%c6)\n1:\n\ts_waitcnt vmcnt(%c5)"
               : "+v"(a[0]), "+v"(a[1]), "+v"(a[2]), "+v"(a[3]) : "s"(mode), "i"(N0), "i"(N1) : "memory", "scc");
}
template <int N0, int N1> __device__ __forceinline__ void waitA2m(v4i (&a)[4], v4i (&b)[4], int mode) {
  asm volatile("s_cmp_eq_u32 %8, 0\n\ts_cbranch_scc1 1f\n\ts_waitcnt vmcnt(%c10)\n1:\n\ts_waitcnt vmcnt(%c9)"
               : "+v"(a[0]), "+v"(a[1]), "+v"(a[2]), "+v"(a[3]), "+v"(b[0]), "+v"(b[1]), "+v"(b[2]), "+v"(b[3])
               : "s"(mode), "i"(N0), "i"(N1) : "memory", "scc");
}
template <int N> __device__ __forceinline__ void waitB(v2i (&b)[4]) {
  asm volatile("s_waitcnt lgkmcnt(%c4)" : "+v"(b[0]), "+v"(b[1]), "+v"(b[2]), "+v"(b[3]) : "i"(N) : "memory");
}
template <int N> __device__ __forceinline__ void waitB3(v2i (&b)[4], v2i (&c)[4], v2i (&d)[4]) {
  asm volatile("s_waitcnt lgkmcnt(%c12)" : "+v"(b[0]), "+v"(b[1]), "+v"(b[2]), "+v"(b[3]), "+v"(c[0]), "+v"(c[1]), "+v"(c[2]), "+v"(c[3]),
               "+v"(d[0]), "+v"(d[1]), "+v"(d[2]), "+v"(d[3]) : "i"(N) : "memory");
}
__device__ __forceinline__ unsigned umin3(unsigned a, unsigned b, unsigned c) { return min(min(a, b), c); }
__device__ __forceinline__ bool missing(const v4i (&a)[4]) {
  unsigned m0 = umin3(a[0].x, a[0].y, a[0].z), m1 = umin3(a[0].w, a[1].x, a[1].y);
  m0 = umin3(m0, a[1].z, a[1].w); m1 = umin3(m1, a[2].x, a[2].y);
  m0 = umin3(m0, a[2].z, a[2].w); m1 = umin3(m1, a[3].x, a[3].y);
  m0 = umin3(m0, a[3].z, a[3].w);
  return __builtin_amdgcn_ballot_w64(min(m0, m1) == 0u) != 0ull;
}

__device__ __forceinline__ int slice_of(int s, int cw, int rank, int mtile) {
  if (s < 2) return rank * 8 + 2 * cw + s;
  int q = (s - 2) * 4 + cw;
  int j = (q * 37) >> 8;
  int i = q - j * 7;
  int src = (rank + 1 + i) & 7;
  return src * 8 + ((j + ROTM * mtile + ROTS * src) & 7);
}

#define ST(i)

struct CW {
  v16f acc[2][4];
  v4i A[4][4];
  v2i B[4][4];
  unsigned baddr[4], wv0, wv1, av, lds0, faddr;
  const uint8_t* Wn; const uint8_t* xb;
  int cw, rank, mode, dead, mtile;
  unsigned long long t0;
  __device__ __forceinline__ const uint8_t* wbase(int sl) const { return Wn + (size_t)slice_of(sl, cw, rank, mtile) * (64 * 4096); }
  __device__ __forceinline__ const uint8_t* abase(int s) const { return xb + (size_t)slice_of(s, cw, rank, mtile) * (8 * 4096); }
  template <int P> __device__ __forceinline__ void pair() {
    dma2(lds0 + ((P >> 2) & 3) * 8192 + (P & 3) * 2048, wv0, wv1, wbase(P >> 2) + (P & 3) * (16 * 4096));
  }
  template <int S, int G, int AI> __device__ __forceinline__ void aissue() {
    if constexpr (AI < 16) {
      if constexpr (AST[AI] == S && AGP[AI] == G) loadA(A[AI & 3], av, abase(AI));
      aissue<S, G, AI + 1>();
    }
  }
  template <int S, int G> __device__ __forceinline__ void gap() {
    aissue<S, G, 2>();
    constexpr int lo = gap_lo(S, G), hi = gap_lo(S, G + 1);
    if constexpr (hi - lo >= 1) pair<lo>();
    if constexpr (hi - lo >= 2) pair<lo + 1>();
    static_assert(hi - lo <= 2, "");
  }
  template <int S, int I, int T, int F> __device__ __forceinline__ void mf() {
    constexpr int U = S & 3;
    acc[I][T] = __builtin_amdgcn_mfma_scale_f32_32x32x64_f8f6f4(cat(A[U][2 * I], A[U][2 * I + 1]), cat4(B[F]), acc[I][T], 0, 0, 0, 0, 0, 0);
    __builtin_amdgcn_sched_barrier(0);
  }
  template <int S, int N0, int N1, bool LOOK> __device__ __forceinline__ void ensure() {
    constexpr int U = S & 3, U1 = (S + 1) & 3;
    if constexpr (LOOK) waitA2m<N0, N1>(A[U], A[U1], mode); else waitA1m<N0, N1>(A[U], mode);
    int look = mode;
    if (__builtin_expect(missing(A[U]), 0)) {
      do {
        if (dead) break;
        __builtin_amdgcn_s_sleep(POLL_SLEEP);
        loadA(A[U], av, abase(S));
        if constexpr (LOOK) waitAA<0>(A[U], A[U1]); else waitA<0>(A[U]);
        if (__builtin_amdgcn_s_memrealtime() - t0 > DEADLINE_TICKS) dead = 1;
      } while (missing(A[U]));
      look = dead ? 0 : 1;
    }
    mode = 0;
    if constexpr (LOOK) {
      if (__builtin_expect(look, 0)) {
        if (missing(A[U1])) { loadA(A[U1], av, abase(S + 1)); mode = 1; }
      }
    }
  }
  template <int S> __device__ __forceinline__ void step() {
    constexpr int BOFF = ((S + 1) & 3) * 8192;
    if constexpr (S < 2) waitA<vm_normal(S)>(A[S]);
    else ensure<S, vm_normal(S), vm_mode(S), can_look(S)>();
    ST(2 + S);
    if constexpr (S == ZSTEP) { if (cw == 0) asm volatile("ds_write_b32 %0, %1 offset:80" :: "v"(faddr), "v"(1) : "memory"); }
    aissue<S, -1, 2>();
    __builtin_amdgcn_sched_barrier(0);
    waitB<12>(B[0]);
    mf<S, 0, 0, 0>(); mf<S, 1, 0, 0>();
    if constexpr (S < 13) readB<BOFF>(B[0], baddr[0]); else readB<2 * 8192>(B[0], baddr[0]);
    waitB3<4>(B[1], B[2], B[3]);
    gap<S, 0>();
    mf<S, 0, 1, 1>();
    gap<S, 1>();
    mf<S, 1, 1, 1>();
    if constexpr (S < 13) readB<BOFF>(B[1], baddr[1]); else readB<3 * 8192>(B[1], baddr[0]);
    gap<S, 2>();
    mf<S, 0, 2, 2>();
    gap<S, 3>();
    mf<S, 1, 2, 2>();
    if constexpr (S < 13) readB<BOFF>(B[2], baddr[2]); else readB<2 * 8192>(B[2], baddr[1]);
    gap<S, 4>();
    mf<S, 0, 3, 3>();
    gap<S, 5>();
    mf<S, 1, 3, 3>();
    if constexpr (S < 13) readB<BOFF>(B[3], baddr[3]); else readB<3 * 8192>(B[3], baddr[1]);
    aissue<S, 6, 2>();
    __builtin_amdgcn_sched_barrier(0);
  }
  template <int T> __device__ __forceinline__ void flag() {
    asm volatile("ds_write_b32 %0, %1 offset:%c2" :: "v"(faddr), "v"(1 + dead), "i"(T * 16) : "memory");
    __builtin_amdgcn_sched_barrier(0);
  }
  template <int I, int T> __device__ __forceinline__ void dump() {
    dump16<T * 8192 + I * 4096>(acc[I][T]);
    __builtin_amdgcn_sched_barrier(0);
  }
  __device__ __forceinline__ void tail() {
    ensure<14, vm_normal(14), vm_mode(14), true>();
    ST(16);
    asm volatile("s_mov_b32 m0, %0" :: "s"(lds0) : "memory");
    __builtin_amdgcn_sched_barrier(0);
    waitB<12>(B[0]);
    mf<14, 0, 0, 0>(); mf<14, 1, 0, 0>();
    readB<2 * 8192>(B[0], baddr[2]);
    ensure<15, 0, 0, false>();
    ST(17);
    waitB<12>(B[1]);
    mf<15, 0, 0, 1>(); mf<15, 1, 0, 1>();
    readB<3 * 8192>(B[1], baddr[2]);
    waitB<12>(B[2]);
    mf<14, 0, 1, 2>();
    dump<0, 0>();
    mf<14, 1, 1, 2>();
    readB<2 * 8192>(B[2], baddr[3]);
    dump<1, 0>(); flag<0>();
    waitB<15>(B[3]);
    mf<15, 0, 1, 3>(); mf<15, 1, 1, 3>();
    readB<3 * 8192>(B[3], baddr[3]);
    waitB<15>(B[0]);
    mf<14, 0, 2, 0>();
    dump<0, 1>();
    mf<14, 1, 2, 0>();
    dump<1, 1>(); flag<1>();
    waitB3<15>(B[1], B[2], B[3]);
    mf<15, 0, 2, 1>(); mf<15, 1, 2, 1>();
    mf<14, 0, 3, 2>();
    dump<0, 2>();
    mf<14, 1, 3, 2>();
    dump<1, 2>(); flag<2>();
    mf<15, 0, 3, 3>(); mf<15, 1, 3, 3>();
    asm volatile("s_nop 7" ::: "memory");
    dump<0, 3>();
    asm volatile("s_nop 7" ::: "memory");
    dump<1, 3>(); flag<3>();
  }
};

__device__ __forceinline__ int wait_group(unsigned faddr, unsigned long long t0) {
  for (unsigned it = 1;; it++) {
    v4i f;
    asm volatile("ds_read_b128 %0, %1\n\ts_waitcnt lgkmcnt(0)" : "=v"(f) : "v"(faddr) : "memory");
    int mn = __builtin_amdgcn_readfirstlane(min(min(f.x, f.y), min(f.z, f.w)));
    if (mn) return __builtin_amdgcn_readfirstlane(f.x | f.y | f.z | f.w);
    __builtin_amdgcn_s_sleep(GSLEEP);
    if ((it & 255) == 0 && __builtin_amdgcn_s_memrealtime() - t0 > 2 * DEADLINE_TICKS) return 2;
  }
}
__device__ __forceinline__ v4f silu4(v4f v, float sc, float k1) {
  const float lim = 60.0f * 1.4426950408889634f;
  v4f u = v * k1, y = v * sc, r;
#pragma unroll
  for (int k = 0; k < 4; k++) u[k] = __builtin_amdgcn_exp2f(__builtin_amdgcn_fmed3f(u[k], -lim, lim));
  u += 1.0f;
#pragma unroll
  for (int k = 0; k < 4; k++) r[k] = __builtin_amdgcn_rcpf(u[k]);
  return y * r;
}
template <int T, int NU> __device__ __forceinline__ void epi(const uint8_t* smem, int u0, int lane, unsigned ovoff, float* obase, float sc) {
  const float k1 = -sc * 1.4426950408889634f;
  const uint8_t* p = smem + T * 8192 + u0 * 1024 + lane * 16;
  v4f part[NU][4];
#pragma unroll
  for (int n = 0; n < NU; n++)
#pragma unroll
    for (int k = 0; k < 4; k++) part[n][k] = *(const v4f*)(p + n * 4096 + k * 32768);
#pragma unroll
  for (int n = 0; n < NU; n++) {
    v4f r = silu4((part[n][0] + part[n][1]) + (part[n][2] + part[n][3]), sc, k1);
    asm volatile("global_store_dwordx4 %0, %1, %2 offset:%c3 sc1\n\ts_nop 1"
                 :: "v"(ovoff), "v"(r), "s"(obase + (size_t)(u0 + 4 * n) * (8 * 4096)), "i"(T * 128) : "memory");
  }
}

__global__ __launch_bounds__(512) void a2a_gemm_kernel(Args a) {
  extern __shared__ __attribute__((aligned(16))) uint8_t smem[];
  const int b = blockIdx.x;
  const int lane = threadIdx.x & 63;
  const int w = __builtin_amdgcn_readfirstlane(threadIdx.x >> 6);
  const int xcd = b & 7, idx = b >> 3;
  const int ntile = ((xcd + 2 * (idx & 3)) & 7) + 8 * (idx & 3);
  const int mtile = idx >> 2;
  const int rank = a.rank;
  uint8_t* const myws = a.myws;
  unsigned* const ctr = (unsigned*)(myws + WS_CTR + (size_t)b * 128);
  unsigned call; float sxv, swv;
  asm volatile("" :: "s"(a.x), "s"(a.W), "s"(a.sx), "s"(a.sw), "s"(a.out), "s"(myws), "s"(a.slow), "s"(rank));
#define LOAD_SCALARS() asm volatile("s_load_dword %0, %3, 0x0\n\ts_load_dword %1, %4, 0x0\n\ts_load_dword %2, %5, 0x0\n\ts_waitcnt lgkmcnt(0)" \
               : "=&s"(call), "=&s"(sxv), "=&s"(swv) : "s"(ctr), "s"(a.sx), "s"(a.sw) : "memory")
  const unsigned long long t0 = __builtin_amdgcn_s_memrealtime();
  const unsigned fbase = (unsigned)(uintptr_t)smem + LDS_FLAGS;
  const unsigned ovoff = (unsigned)((((lane >> 3) & 1) * 4 + (lane >> 4)) * 16384 + (lane & 7) * 16);
  float* const obase = a.out + (size_t)(mtile * 64) * 4096 + ntile * 128;

  if (w < 4) {
    __builtin_amdgcn_s_setprio(3);
    const int pl = JMAP(mtile), pm = ntile >> 2, pr = ntile & 3;
    const int pj = (pl + ROTM * pm + ROTS * rank) & 7;
    const unsigned d0 = (unsigned)(rank + 1 + w) & 7u, d1 = (unsigned)(rank + 5 + w) & 7u;
    const unsigned stage = (unsigned)(uintptr_t)smem + LDS_STAGE + w * 2048;
    const uint8_t* src = a.x + (size_t)(pm * 64 + pr * 16) * 512 + pj * 64;
    const unsigned sv = (unsigned)((lane >> 2) * 512 + (lane & 3) * 16);
    asm volatile("s_mov_b32 m0, %0\n\ts_nop 0\n\tglobal_load_lds_dwordx4 %1, %2" :: "s"(stage), "v"(sv), "s"(src + (size_t)d0 * (512 * 512)) : "memory");
    if (w < 3) asm volatile("s_mov_b32 m0, %0\n\ts_nop 0\n\tglobal_load_lds_dwordx4 %1, %2" :: "s"(stage + 1024), "v"(sv), "s"(src + (size_t)d1 * (512 * 512)) : "memory");
    if (w == 0) asm volatile("ds_write_b32 %0, %1\n\ts_waitcnt lgkmcnt(0)" :: "v"(fbase + (lane & 31) * 4), "v"(0) : "memory");
    __builtin_amdgcn_s_barrier();
    uint8_t* const ws0 = a.peer[w];
    uint8_t* const ws1 = a.peer[w + 4];
    LOAD_SCALARS();
    asm volatile("ds_write_b32 %0, %1 offset:64" :: "v"(fbase + w * 4), "v"(1) : "memory");
    const size_t doff = WS_XBUF + (size_t)(call % NBUF) * XBUF_BYTES + (size_t)(rank * 64 + pj * 8 + pm) * 4096 + (pr >> 1) * 2048 + (pr & 1) * 256;
    const unsigned so = (unsigned)(((lane >> 4) & 1) * 1024 + (lane >> 5) * 512 + (lane & 15) * 16);
    const unsigned ra = stage + (lane & 15) * 64 + (lane >> 4) * 16;
    v4i v0, v1;
    asm volatile("s_waitcnt vmcnt(0)\n\tds_read_b128 %0, %2\n\tds_read_b128 %1, %2 offset:1024\n\ts_waitcnt lgkmcnt(0)"
                 : "=&v"(v0), "=&v"(v1) : "v"(ra) : "memory");
    v0.x = v0.x ? v0.x : 0x80; v0.y = v0.y ? v0.y : 0x80; v0.z = v0.z ? v0.z : 0x80; v0.w = v0.w ? v0.w : 0x80;
    v1.x = v1.x ? v1.x : 0x80; v1.y = v1.y ? v1.y : 0x80; v1.z = v1.z ? v1.z : 0x80; v1.w = v1.w ? v1.w : 0x80;
    const int slot = pl * 32 + ntile - SLOT0;
    if (slot > 0) {
      const unsigned long long due = t0 + TBASE + (((unsigned)slot * SLOT8) >> 3);
      while (__builtin_amdgcn_s_memrealtime() < due) __builtin_amdgcn_s_sleep(1);
    }
    asm volatile("global_store_dwordx4 %0, %1, %2 sc0 sc1\n\ts_nop 1" :: "v"(so), "v"(v0), "s"(ws0 + doff) : "memory");
    if (w < 3) asm volatile("global_store_dwordx4 %0, %1, %2 sc0 sc1\n\ts_nop 1" :: "v"(so), "v"(v1), "s"(ws1 + doff) : "memory");
    __builtin_amdgcn_s_setprio(0);
    for (;;) {
      unsigned v;
      asm volatile("ds_read_b32 %0, %1 offset:80\n\ts_waitcnt lgkmcnt(0)" : "=v"(v) : "v"(fbase) : "memory");
      if (__builtin_amdgcn_readfirstlane(v)) break;
      if (__builtin_amdgcn_s_memrealtime() - t0 > 2 * DEADLINE_TICKS) break;
      __builtin_amdgcn_s_sleep(4);
    }
    {
      uint8_t* z = myws + WS_XBUF + (size_t)((call + 2u) % NBUF) * XBUF_BYTES + (size_t)b * 1024;
      const v4i zero = {0, 0, 0, 0};
      const unsigned zo = lane * 16;
      asm volatile("global_store_dwordx4 %0, %1, %2 sc1\n\ts_nop 1" :: "v"(zo), "v"(zero), "s"(z + (size_t)d0 * 262144) : "memory");
      if (w < 3) asm volatile("global_store_dwordx4 %0, %1, %2 sc1\n\ts_nop 1" :: "v"(zo), "v"(zero), "s"(z + (size_t)d1 * 262144) : "memory");
    }
    const float sc0 = sxv * swv;
    float sc;
    sc = (wait_group(fbase, t0) & 2) ? __builtin_nanf("") : sc0;
    if (w == 0) {
      (void)wait_group(fbase + 64, t0);
      if (lane == 0) __hip_atomic_store(ctr, call + 1u, __ATOMIC_RELAXED, __HIP_MEMORY_SCOPE_AGENT);
    }
    if constexpr (HU[0] > 0) epi<0, HU[0] / 4>(smem, w, lane, ovoff, obase, sc);
    if constexpr (HU[1] > 0) { sc = (wait_group(fbase + 16, t0) & 2) ? __builtin_nanf("") : sc0; epi<1, HU[1] / 4>(smem, w, lane, ovoff, obase, sc); }
    if constexpr (HU[2] > 0) { sc = (wait_group(fbase + 32, t0) & 2) ? __builtin_nanf("") : sc0; epi<2, HU[2] / 4>(smem, w, lane, ovoff, obase, sc); }
    if constexpr (HU[3] > 0) { sc = (wait_group(fbase + 48, t0) & 2) ? __builtin_nanf("") : sc0; epi<3, HU[3] / 4>(smem, w, lane, ovoff, obase, sc); }
  } else {
    CW c;
    c.cw = w - 4; c.rank = rank; c.mtile = mtile; c.mode = 0; c.dead = 0;
    c.t0 = t0;
    c.lds0 = (unsigned)(uintptr_t)smem + c.cw * 32768;
    c.faddr = fbase + c.cw * 4;
    c.wv0 = (unsigned)((lane >> 3) * 4096 + (((lane & 7) ^ (((lane >> 4) & 3) << 1)) << 4));
    c.wv1 = c.wv0 + 8 * 4096 - 1024;
    c.Wn = a.W + ntile * 128;
    c.av = lane * 16;
    {
      const int dlo = (a.slow - xcd) & 7;
      if ((dlo & 1) == 0 && c.cw < 2)
        asm volatile("s_mov_b32 m0, %0\n\ts_nop 0\n\tglobal_load_lds_dword %1, %2"
                     :: "s"((unsigned)(uintptr_t)smem + LDS_JUNK), "v"(c.av << 8),
                        "s"(a.W + (a.slow + 8 * (dlo >> 1)) * 128 + (size_t)(idx * 128 + c.cw * 64) * 4096) : "memory");
    }
    c.pair<0>(); c.pair<1>(); c.pair<2>(); c.pair<3>();
    __builtin_amdgcn_s_barrier();
    {
      int g = lane >> 4, h = g >> 1, cg = g & 1, i = lane & 15, q = i >> 1, p = i & 1;
      int sw = ((q >> 1) & 3) << 1;
#pragma unroll
      for (int t = 0; t < 4; t++) c.baddr[t] = c.lds0 + (32 * h + q) * 128 + (((2 * t + cg) ^ sw) << 4) + 8 * p;
    }
#pragma unroll
    for (int i = 0; i < 2; i++)
#pragma unroll
      for (int t = 0; t < 4; t++) c.acc[i][t] = (v16f){};
    const unsigned stage = c.lds0 + 3 * 8192;
    {
      const unsigned xve = (unsigned)((lane >> 3) * 512 + (((lane & 7) ^ ((lane >> 4) & 3)) << 4));
      const unsigned xvo = (xve ^ 64u) + 8 * 512 - 1024;
      const uint8_t* xo = a.x + (size_t)(rank * 512 + mtile * 64) * 512 + c.cw * 128;
      dma2(stage, xve, xvo, xo); dma2(stage + 2048, xve, xvo, xo + 16 * 512);
      dma2(stage + 4096, xve, xvo, xo + 32 * 512); dma2(stage + 6144, xve, xvo, xo + 48 * 512);
    }
    c.pair<4>(); c.pair<5>(); c.pair<6>(); c.pair<7>();
    if constexpr (WEND[0] > 8) { c.pair<8>(); } if constexpr (WEND[0] > 9) { c.pair<9>(); }
    if constexpr (WEND[0] > 10) { c.pair<10>(); } if constexpr (WEND[0] > 11) { c.pair<11>(); }
    LOAD_SCALARS();
    c.xb = myws + WS_XBUF + (size_t)(call % NBUF) * XBUF_BYTES + (size_t)mtile * 4096;
    asm volatile("s_waitcnt vmcnt(%c0)" :: "i"(SC.tot[0] - SC.posA[0]) : "memory");
    {
      const unsigned xr0 = stage + (unsigned)((lane & 31) * 128), xs = (unsigned)(((2 * (lane >> 5)) ^ ((lane >> 1) & 7)) << 4);
      readX(c.A[0], xr0 + xs, xr0 + (xs ^ 16u));
      readX(c.A[1], xr0 + (xs ^ 64u), xr0 + (xs ^ 80u));
      waitX(c.A[0], c.A[1]);
    }
    readB<0>(c.B[0], c.baddr[0]); readB<0>(c.B[1], c.baddr[1]); readB<0>(c.B[2], c.baddr[2]); readB<0>(c.B[3], c.baddr[3]);
    c.step<0>(); c.step<1>(); c.step<2>(); c.step<3>(); c.step<4>(); c.step<5>(); c.step<6>();
    c.step<7>(); c.step<8>(); c.step<9>(); c.step<10>(); c.step<11>(); c.step<12>(); c.step<13>();
    c.tail();
    const float sc0 = sxv * swv;
    float sc;
    if constexpr (HU[0] < 8) { sc = (wait_group(fbase, t0) & 2) ? __builtin_nanf("") : sc0; epi<0, (8 - HU[0]) / 4>(smem, HU[0] + c.cw, lane, ovoff, obase, sc); }
    if constexpr (HU[1] < 8) { sc = (wait_group(fbase + 16, t0) & 2) ? __builtin_nanf("") : sc0; epi<1, (8 - HU[1]) / 4>(smem, HU[1] + c.cw, lane, ovoff, obase, sc); }
    if constexpr (HU[2] < 8) { sc = (wait_group(fbase + 32, t0) & 2) ? __builtin_nanf("") : sc0; epi<2, (8 - HU[2]) / 4>(smem, HU[2] + c.cw, lane, ovoff, obase, sc); }
    if constexpr (HU[3] < 8) {
      sc = (wait_group(fbase + 48, t0) & 2) ? __builtin_nanf("") : sc0;
      epi<3, (8 - HU[3]) / 4>(smem, HU[3] + c.cw, lane, ovoff, obase, sc);
    }
  }
}

extern "C" void kernel_launch(int rank, int nranks, ncclComm_t comm, hipStream_t stream,
                              void* const* d_in, const int* in_sizes, int n_in,
                              void* d_out, int out_size, const int* in_dtypes, int out_dtype,
                              void* const* d_out_all, void* const* d_ws_all, size_t ws_bytes) {
  if (nranks != NRANKS || ws_bytes < WS_NEED || (unsigned)rank >= (unsigned)NRANKS) {
    fprintf(stderr, "unsupported configuration\n");
    return;
  }
  static bool init = false;
  if (!init) {
    HIP_CHECK(hipFuncSetAttribute((const void*)a2a_gemm_kernel, hipFuncAttributeMaxDynamicSharedMemorySize, LDS_BYTES));
    init = true;
  }
  Args a;
  a.x = (const uint8_t*)d_in[0]; a.W = (const uint8_t*)d_in[1];
  a.sx = (const float*)d_in[2]; a.sw = (const float*)d_in[3];
  a.out = (float*)d_out;
  a.myws = (uint8_t*)d_ws_all[rank];
  for (int p = 0; p < NRANKS; p++) a.peer[p] = (uint8_t*)d_ws_all[(rank + 1 + p) % NRANKS];
  a.rank = rank; a.slow = (int)((3u - (unsigned)((uintptr_t)d_in[1] >> 7)) & 7u);
  a2a_gemm_kernel<<<GRID, 512, LDS_BYTES, stream>>>(a);
}
```

```cpp
#include <hip/hip_runtime.h>
#include <rccl/rccl.h>
#include <cstdio>
#include <cstdint>

#define HIP_CHECK(expr) do { hipError_t _e = (expr); if (_e != hipSuccess) { \
    fprintf(stderr, "HIP error %s at %s:%d\n", hipGetErrorString(_e), __FILE__, __LINE__); \
    return; } } while (0)

typedef int v2i __attribute__((ext_vector_type(2)));
typedef int v4i __attribute__((ext_vector_type(4)));
typedef int v8i __attribute__((ext_vector_type(8)));
typedef float v4f __attribute__((ext_vector_type(4)));
typedef float v16f __attribute__((ext_vector_type(16)));

#define SLOT8 20
#define SLOT0 24
#define TBASE 60
#define JMAP(mt) (7 - (mt))
#define ZSTEP 12
#define POLL_SLEEP 2
#define GSLEEP 1
#define BMASK 0x400
#define CPRIO 0
#define ROTM 1
#define ROTS 1
#define HU_TABLE {8, 8, 8, 0}

constexpr int NRANKS = 8;
constexpr int GRID = 256;
constexpr int LDS_STAGE = 131072;
constexpr int LDS_FLAGS = LDS_STAGE + 8192;
constexpr int LDS_JUNK = LDS_FLAGS + 128;
constexpr int LDS_BYTES = LDS_JUNK + 256;
constexpr size_t WS_CTR = 0;
constexpr size_t WS_DBG = 256 << 10;
constexpr size_t WS_XBUF = 1 << 20;
constexpr unsigned NBUF = 8;
constexpr size_t XBUF_BYTES = 2 << 20;
constexpr size_t WS_NEED = WS_XBUF + NBUF * XBUF_BYTES;
#define DEADLINE 50000000ull
constexpr unsigned long long DEADLINE_TICKS = DEADLINE;

struct Args {
  const uint8_t* x; const uint8_t* W; const float* sx; const float* sw; float* out;
  uint8_t* myws;
  int rank, slow;
  uint8_t* peer[NRANKS];
};

#define WEND_TABLE {12, 20, 24, 28, 32, 36, 40, 44, 48, 52, 56, 60, 64, 64}
#define AST_TABLE {0, 0, 1, 1, 2, 3, 4, 5, 6, 7, 8, 9, 10, 11, 12, 12}
#define AGP_TABLE {0, 0, 3, 3, -1, -1, -1, -1, -1, -1, -1, -1, -1, -1, 0, 2}
constexpr int WEND[14] = WEND_TABLE;
constexpr int AST[16] = AST_TABLE;
constexpr int AGP[16] = AGP_TABLE;
constexpr int HU[4] = HU_TABLE;
constexpr int gap_lo(int s, int g) {
  int n = WEND[s + 1] - WEND[s];
  return WEND[s] + (n * g + 5) / 6;
}
constexpr int cmin(int a, int b) { return a < b ? a : b; }
constexpr int cmax(int a, int b) { return a > b ? a : b; }
struct Sched {
  int posA[16], posW[16], tot[14], mid[8];
  constexpr Sched() : posA(), posW(), tot(), mid() {
    int n = 0;
    n += 8; posW[0] = n;
    n += 8; posA[0] = posA[1] = n;
    for (int p = 4; p < WEND[0]; p++) { n += 2; if ((p & 3) == 3) posW[p >> 2] = n; }
    for (int s = 0; s < 13; s++) {
      tot[s] = n;
      for (int g = -1; g <= 6; g++) {
        if (s == 12 && g >= 0) mid[g] = n;
        for (int a = 2; a < 16; a++) if (AST[a] == s && AGP[a] == g) { n += 4; posA[a] = n; }
        if (g >= 0 && g < 6) for (int p = gap_lo(s, g); p < gap_lo(s, g + 1); p++) { n += 2; if ((p & 3) == 3) posW[p >> 2] = n; }
      }
    }
    tot[13] = n;
  }
};
constexpr Sched SC;
constexpr int need_normal(int s) { return s >= 13 ? SC.posA[s] : cmax(SC.posA[s], SC.posW[s + 1]); }
constexpr int vm_normal(int s) { return cmin(63, SC.tot[cmin(s, 13)] - need_normal(s)); }
constexpr bool can_look(int s) { return s < 15 && AST[s + 1] < s; }
constexpr int vm_mode(int s) {
  return (s < 3 || s > 13) ? 0 : cmin(63, SC.tot[s] - cmax(need_normal(s), cmax(SC.tot[s - 1], can_look(s) ? SC.posA[s + 1] : 0)));
}
constexpr bool sched_ok() {
  if (WEND[0] < 8 || WEND[0] > 12 || WEND[13] != 64) return false;
  for (int s = 0; s < 13; s++) {
    if (WEND[s + 1] < WEND[s] || WEND[s + 1] - WEND[s] > 8) return false;
    if (WEND[s + 1] > 4 * (s + 5)) return false;
    if (WEND[s] < 4 * (s + 2)) return false;
    if (vm_normal(s) < 0 || vm_mode(s) < 0) return false;
  }
  if (SC.mid[4] < SC.posW[14] || SC.mid[6] < SC.posW[15]) return false;
  for (int a = 2; a < 16; a++) {
    if (AST[a] >= cmin(a, 13) || AST[a] < a - 3 || AST[a] < 0) return false;
    if (a > 2 && SC.posA[a] < SC.posA[a - 1]) return false;
  }
  return true;
}
static_assert(sched_ok(), "bad schedule");

__device__ __forceinline__ v8i cat(v4i a, v4i b) { return __builtin_shufflevector(a, b, 0, 1, 2, 3, 4, 5, 6, 7); }
__device__ __forceinline__ v8i cat4(const v2i (&b)[4]) {
  v4i x = __builtin_shufflevector(b[0], b[1], 0, 1, 2, 3), y = __builtin_shufflevector(b[2], b[3], 0, 1, 2, 3);
  return cat(x, y);
}
__device__ __forceinline__ void loadA(v4i (&a)[4], unsigned voff, const uint8_t* base) {
  asm volatile("global_load_dwordx4 %0, %4, %5 sc1\n\t"
               "global_load_dwordx4 %1, %4, %5 offset:1024 sc1\n\t"
               "global_load_dwordx4 %2, %4, %5 offset:2048 sc1\n\t"
               "global_load_dwordx4 %3, %4, %5 offset:3072 sc1"
               : "=&v"(a[0]), "=&v"(a[1]), "=&v"(a[2]), "=&v"(a[3]) : "v"(voff), "s"(base) : "memory");
}
__device__ __forceinline__ void dma2(unsigned m0v, unsigned voff0, unsigned voff1, const uint8_t* base) {
  asm volatile("s_mov_b32 m0, %0\n\ts_nop 0\n\t"
               "global_load_lds_dwordx4 %1, %3\n\t"
               "global_load_lds_dwordx4 %2, %3 offset:1024"
               :: "s"(m0v), "v"(voff0), "v"(voff1), "s"(base) : "memory");
}
__device__ __forceinline__ void readX(v4i (&a)[4], unsigned a0, unsigned a1) {
  asm volatile("ds_read_b128 %0, %4\n\tds_read_b128 %1, %5\n\tds_read_b128 %2, %4 offset:4096\n\tds_read_b128 %3, %5 offset:4096"
               : "=&v"(a[0]), "=&v"(a[1]), "=&v"(a[2]), "=&v"(a[3]) : "v"(a0), "v"(a1) : "memory");
}
__device__ __forceinline__ void waitX(v4i (&a)[4], v4i (&b)[4]) {
  asm volatile("s_waitcnt lgkmcnt(0)" : "+v"(a[0]), "+v"(a[1]), "+v"(a[2]), "+v"(a[3]), "+v"(b[0]), "+v"(b[1]), "+v"(b[2]), "+v"(b[3]) :: "memory");
}
template <int OFF, int R> __device__ __forceinline__ void dump8(const v16f& c) {
  asm volatile("ds_write_addtid_b32 %0 offset:%c8\n\tds_write_addtid_b32 %1 offset:%c9\n\tds_write_addtid_b32 %2 offset:%c10\n\tds_write_addtid_b32 %3 offset:%c11\n\t"
               "ds_write_addtid_b32 %4 offset:%c12\n\tds_write_addtid_b32 %5 offset:%c13\n\tds_write_addtid_b32 %6 offset:%c14\n\tds_write_addtid_b32 %7 offset:%c15"
               :: "v"(c[R]), "v"(c[R + 1]), "v"(c[R + 2]), "v"(c[R + 3]), "v"(c[R + 4]), "v"(c[R + 5]), "v"(c[R + 6]), "v"(c[R + 7]),
                  "i"(OFF), "i"(OFF + 256), "i"(OFF + 512), "i"(OFF + 768), "i"(OFF + 1024), "i"(OFF + 1280), "i"(OFF + 1536), "i"(OFF + 1792) : "memory");
}
template <int OFF> __device__ __forceinline__ void readB(v2i (&b)[4], unsigned addr) {
  asm volatile("ds_read_b64_tr_b8 %0, %4 offset:%c5\n\t"
               "ds_read_b64_tr_b8 %1, %4 offset:%c6\n\t"
               "ds_read_b64_tr_b8 %2, %4 offset:%c7\n\t"
               "ds_read_b64_tr_b8 %3, %4 offset:%c8"
               : "=&v"(b[0]), "=&v"(b[1]), "=&v"(b[2]), "=&v"(b[3])
               : "v"(addr), "i"(OFF), "i"(OFF + 1024), "i"(OFF + 2048), "i"(OFF + 3072) : "memory");
}
template <int N> __device__ __forceinline__ void waitA(v4i (&a)[4]) {
  asm volatile("s_waitcnt vmcnt(%c4)" : "+v"(a[0]), "+v"(a[1]), "+v"(a[2]), "+v"(a[3]) : "i"(N) : "memory");
}
template <int N> __device__ __forceinline__ void waitAA(v4i (&a)[4], v4i (&b)[4]) {
  asm volatile("s_waitcnt vmcnt(%c8)" : "+v"(a[0]), "+v"(a[1]), "+v"(a[2]), "+v"(a[3]), "+v"(b[0]), "+v"(b[1]), "+v"(b[2]), "+v"(b[3]) : "i"(N) : "memory");
}
template <int N0, int N1> __device__ __forceinline__ void waitA1m(v4i (&a)[4], int mode) {
  asm volatile("s_cmp_eq_u32 %4, 0\n\ts_cbranch_scc1 1f\n\ts_waitcnt vmcnt(%c6)\n1:\n\ts_waitcnt vmcnt(%c5)"
               : "+v"(a[0]), "+v"(a[1]), "+v"(a[2]), "+v"(a[3]) : "s"(mode), "i"(N0), "i"(N1) : "memory", "scc");
}
template <int N0, int N1> __device__ __forceinline__ void waitA2m(v4i (&a)[4], v4i (&b)[4], int mode) {
  asm volatile("s_cmp_eq_u32 %8, 0\n\ts_cbranch_scc1 1f\n\ts_waitcnt vmcnt(%c10)\n1:\n\ts_waitcnt vmcnt(%c9)"
               : "+v"(a[0]), "+v"(a[1]), "+v"(a[2]), "+v"(a[3]), "+v"(b[0]), "+v"(b[1]), "+v"(b[2]), "+v"(b[3])
               : "s"(mode), "i"(N0), "i"(N1) : "memory", "scc");
}
template <int N> __device__ __forceinline__ void waitB(v2i (&b)[4]) {
  asm volatile("s_waitcnt lgkmcnt(%c4)" : "+v"(b[0]), "+v"(b[1]), "+v"(b[2]), "+v"(b[3]) : "i"(N) : "memory");
}
template <int N> __device__ __forceinline__ void waitB3(v2i (&b)[4], v2i (&c)[4], v2i (&d)[4]) {
  asm volatile("s_waitcnt lgkmcnt(%c12)" : "+v"(b[0]), "+v"(b[1]), "+v"(b[2]), "+v"(b[3]), "+v"(c[0]), "+v"(c[1]), "+v"(c[2]), "+v"(c[3]),
               "+v"(d[0]), "+v"(d[1]), "+v"(d[2]), "+v"(d[3]) : "i"(N) : "memory");
}
__device__ __forceinline__ unsigned umin3(unsigned a, unsigned b, unsigned c) { return min(min(a, b), c); }
__device__ __forceinline__ bool missing(const v4i (&a)[4]) {
  unsigned m0 = umin3(a[0].x, a[0].y, a[0].z), m1 = umin3(a[0].w, a[1].x, a[1].y);
  m0 = umin3(m0, a[1].z, a[1].w); m1 = umin3(m1, a[2].x, a[2].y);
  m0 = umin3(m0, a[2].z, a[2].w); m1 = umin3(m1, a[3].x, a[3].y);
  m0 = umin3(m0, a[3].z, a[3].w);
  return __builtin_amdgcn_ballot_w64(min(m0, m1) == 0u) != 0ull;
}

__device__ __forceinline__ int slice_of(int s, int cw, int rank, int mtile) {
  if (s < 2) return rank * 8 + 2 * cw + s;
  int q = (s - 2) * 4 + cw;
  int j = (q * 37) >> 8;
  int i = q - j * 7;
  int src = (rank + 1 + i) & 7;
  return src * 8 + ((j + ROTM * mtile + ROTS * src) & 7);
}

#define ST(i)

struct CW {
  v16f acc[2][4];
  v4i A[4][4];
  v2i B[4][4];
  unsigned baddr[4], wv0, wv1, av, lds0, faddr;
  const uint8_t* Wn; const uint8_t* xb;
  int cw, rank, mode, dead, mtile;
  unsigned long long t0;
  __device__ __forceinline__ const uint8_t* wbase(int sl) const { return Wn + (size_t)slice_of(sl, cw, rank, mtile) * (64 * 4096); }
  __device__ __forceinline__ const uint8_t* abase(int s) const { return xb + (size_t)slice_of(s, cw, rank, mtile) * (8 * 4096); }
  template <int P> __device__ __forceinline__ void pair() {
    dma2(lds0 + ((P >> 2) & 3) * 8192 + (P & 3) * 2048, wv0, wv1, wbase(P >> 2) + (P & 3) * (16 * 4096));
  }
  template <int S, int G, int AI> __device__ __forceinline__ void aissue() {
    if constexpr (AI < 16) {
      if constexpr (AST[AI] == S && AGP[AI] == G) loadA(A[AI & 3], av, abase(AI));
      aissue<S, G, AI + 1>();
    }
  }
  template <int S, int G> __device__ __forceinline__ void gap() {
    aissue<S, G, 2>();
    constexpr int lo = gap_lo(S, G), hi = gap_lo(S, G + 1);
    if constexpr (hi - lo >= 1) pair<lo>();
    if constexpr (hi - lo >= 2) pair<lo + 1>();
    static_assert(hi - lo <= 2, "");
  }
  template <int S, int I, int T, int F> __device__ __forceinline__ void mf() {
    constexpr int U = S & 3;
    acc[I][T] = __builtin_amdgcn_mfma_scale_f32_32x32x64_f8f6f4(cat(A[U][2 * I], A[U][2 * I + 1]), cat4(B[F]), acc[I][T], 0, 0, 0, 0, 0, 0);
    __builtin_amdgcn_sched_barrier(0);
  }
  template <int S, int N0, int N1, bool LOOK> __device__ __forceinline__ void ensure() {
    constexpr int U = S & 3, U1 = (S + 1) & 3;
    if constexpr (LOOK) waitA2m<N0, N1>(A[U], A[U1], mode); else waitA1m<N0, N1>(A[U], mode);
    int look = mode;
    if (__builtin_expect(missing(A[U]), 0)) {
      do {
        if (dead) break;
        __builtin_amdgcn_s_sleep(POLL_SLEEP);
        loadA(A[U], av, abase(S));
        if constexpr (LOOK) waitAA<0>(A[U], A[U1]); else waitA<0>(A[U]);
        if (__builtin_amdgcn_s_memrealtime() - t0 > DEADLINE_TICKS) dead = 1;
      } while (missing(A[U]));
      look = dead ? 0 : 1;
    }
    mode = 0;
    if constexpr (LOOK) {
      if (__builtin_expect(look, 0)) {
        if (missing(A[U1])) { loadA(A[U1], av, abase(S + 1)); mode = 1; }
      }
    }
  }
  template <int S> __device__ __forceinline__ void step() {
    constexpr int BOFF = ((S + 1) & 3) * 8192;
    if constexpr (S < 2) waitA<vm_normal(S)>(A[S]);
    else ensure<S, vm_normal(S), vm_mode(S), can_look(S)>();
    ST(2 + S);
    if constexpr ((BMASK >> S) & 1) __builtin_amdgcn_s_barrier();
    if constexpr (S == ZSTEP) { if (cw == 0) asm volatile("ds_write_b32 %0, %1 offset:80" :: "v"(faddr), "v"(1) : "memory"); }
    aissue<S, -1, 2>();
    __builtin_amdgcn_sched_barrier(0);
    waitB<12>(B[0]);
    mf<S, 0, 0, 0>(); mf<S, 1, 0, 0>();
    readB<BOFF>(B[0], baddr[0]);
    waitB3<4>(B[1], B[2], B[3]);
    gap<S, 0>();
    mf<S, 0, 1, 1>();
    gap<S, 1>();
    mf<S, 1, 1, 1>();
    readB<BOFF>(B[1], baddr[1]);
    gap<S, 2>();
    mf<S, 0, 2, 2>();
    gap<S, 3>();
    mf<S, 1, 2, 2>();
    if constexpr (S < 12) readB<BOFF>(B[2], baddr[2]);
    else { asm volatile("s_waitcnt vmcnt(%c0)" :: "i"(cmin(63, SC.mid[4] - SC.posW[14])) : "memory"); readB<2 * 8192>(B[2], baddr[0]); }
    gap<S, 4>();
    mf<S, 0, 3, 3>();
    gap<S, 5>();
    mf<S, 1, 3, 3>();
    if constexpr (S < 12) readB<BOFF>(B[3], baddr[3]);
    else { asm volatile("s_waitcnt vmcnt(%c0)" :: "i"(cmin(63, SC.mid[6] - SC.posW[15])) : "memory"); readB<3 * 8192>(B[3], baddr[0]); }
    aissue<S, 6, 2>();
    __builtin_amdgcn_sched_barrier(0);
  }
  template <int T> __device__ __forceinline__ void flag() {
    asm volatile("ds_write_b32 %0, %1 offset:%c2" :: "v"(faddr), "v"(1 + dead), "i"(T * 16) : "memory");
    __builtin_amdgcn_sched_barrier(0);
  }
  template <int T, int Q> __device__ __forceinline__ void dump() {
    dump8<T * 8192 + Q * 2048, (Q & 1) * 8>(acc[Q >> 1][T]);
    __builtin_amdgcn_sched_barrier(0);
  }
  __device__ __forceinline__ void tail() {
    ensure<13, vm_normal(13), vm_mode(13), can_look(13)>();
    ST(16);
    asm volatile("s_mov_b32 m0, %0" :: "s"(lds0) : "memory");
    __builtin_amdgcn_sched_barrier(0);
    waitB<12>(B[0]);
    mf<13, 0, 0, 0>(); mf<13, 1, 0, 0>();
    readB<2 * 8192>(B[0], baddr[1]);
    waitB<12>(B[1]);
    mf<13, 0, 1, 1>(); mf<13, 1, 1, 1>();
    readB<3 * 8192>(B[1], baddr[1]);
    ensure<14, vm_normal(14), 0, true>();
    waitB<12>(B[2]);
    mf<14, 0, 0, 2>(); mf<14, 1, 0, 2>();
    readB<1 * 8192>(B[2], baddr[2]);
    ensure<15, 0, 0, false>();
    ST(17);
    waitB<12>(B[3]);
    mf<15, 0, 0, 3>(); mf<15, 1, 0, 3>();
    readB<2 * 8192>(B[3], baddr[2]);
    waitB<8>(B[0]); waitB<8>(B[1]);
    mf<14, 0, 1, 0>(); dump<0, 0>();
    mf<14, 1, 1, 0>();
    readB<3 * 8192>(B[0], baddr[2]);
    dump<0, 1>();
    mf<15, 0, 1, 1>(); dump<0, 2>();
    mf<15, 1, 1, 1>();
    readB<1 * 8192>(B[1], baddr[3]);
    dump<0, 3>(); flag<0>();
    waitB<15>(B[2]); waitB<15>(B[3]);
    mf<13, 0, 2, 2>(); mf<13, 1, 2, 2>();
    waitB<9>(B[1]); waitB<9>(B[0]);
    readB<2 * 8192>(B[2], baddr[3]);
    dump<1, 0>();
    mf<14, 0, 2, 3>(); dump<1, 1>();
    mf<14, 1, 2, 3>();
    readB<3 * 8192>(B[3], baddr[3]);
    dump<1, 2>();
    mf<15, 0, 2, 0>(); dump<1, 3>(); flag<1>();
    mf<15, 1, 2, 0>();
    waitB<15>(B[2]); waitB<15>(B[3]);
    mf<13, 0, 3, 1>(); dump<2, 0>();
    mf<13, 1, 3, 1>(); dump<2, 1>();
    mf<14, 0, 3, 2>(); dump<2, 2>();
    mf<14, 1, 3, 2>(); dump<2, 3>(); flag<2>();
    mf<15, 0, 3, 3>(); mf<15, 1, 3, 3>();
    asm volatile("s_nop 7" ::: "memory");
    dump<3, 0>(); dump<3, 1>();
    asm volatile("s_nop 7" ::: "memory");
    dump<3, 2>(); dump<3, 3>(); flag<3>();
  }
};

__device__ __forceinline__ int wait_group(unsigned faddr, unsigned long long t0) {
  for (unsigned it = 1;; it++) {
    v4i f;
    asm volatile("ds_read_b128 %0, %1\n\ts_waitcnt lgkmcnt(0)" : "=v"(f) : "v"(faddr) : "memory");
    int mn = __builtin_amdgcn_readfirstlane(min(min(f.x, f.y), min(f.z, f.w)));
    if (mn) return __builtin_amdgcn_readfirstlane(f.x | f.y | f.z | f.w);
    __builtin_amdgcn_s_sleep(GSLEEP);
    if ((it & 255) == 0 && __builtin_amdgcn_s_memrealtime() - t0 > 2 * DEADLINE_TICKS) return 2;
  }
}
#define SILU_V 1
__device__ __forceinline__ v4f silu4(v4f v, float sc, float k1) {
  const float lim = 60.0f * 1.4426950408889634f;
  v4f u = v * k1 - lim, y = v * sc, r;
#pragma unroll
  for (int k = 0; k < 4; k++) asm("v_exp_f32_e64 %0, %1 clamp" : "=v"(u[k]) : "v"(u[k]));
  u = u * 1.1420073898156842e26f + 1.0f;
#pragma unroll
  for (int k = 0; k < 4; k++) r[k] = __builtin_amdgcn_rcpf(u[k]);
  return y * r;
}
struct Batch { v4i f; v4f p[8]; };
template <int T> __device__ __forceinline__ void batch_issue(Batch& q, unsigned fbase, unsigned e0, unsigned e1) {
  asm volatile("ds_read_b128 %0, %9 offset:%c12\n\t"
               "ds_read_b128 %1, %10 offset:%c13\n\tds_read_b128 %2, %10 offset:%c14\n\tds_read_b128 %3, %11 offset:%c13\n\tds_read_b128 %4, %11 offset:%c14\n\t"
               "ds_read_b128 %5, %10 offset:%c15\n\tds_read_b128 %6, %10 offset:%c16\n\tds_read_b128 %7, %11 offset:%c15\n\tds_read_b128 %8, %11 offset:%c16"
               : "=&v"(q.f), "=&v"(q.p[0]), "=&v"(q.p[1]), "=&v"(q.p[2]), "=&v"(q.p[3]), "=&v"(q.p[4]), "=&v"(q.p[5]), "=&v"(q.p[6]), "=&v"(q.p[7])
               : "v"(fbase), "v"(e0), "v"(e1), "i"(T * 16), "i"(T * 8192), "i"(T * 8192 + 32768), "i"(T * 8192 + 4096), "i"(T * 8192 + 4096 + 32768) : "memory");
}
__device__ __forceinline__ void batch_wait(Batch& q) {
  asm volatile("s_waitcnt lgkmcnt(0)" : "+v"(q.f), "+v"(q.p[0]), "+v"(q.p[1]), "+v"(q.p[2]), "+v"(q.p[3]), "+v"(q.p[4]), "+v"(q.p[5]), "+v"(q.p[6]), "+v"(q.p[7]) :: "memory");
}
template <int T> __device__ __forceinline__ float batch_finish(Batch& q, unsigned fbase, unsigned e0, unsigned e1, unsigned long long t0, float sc0) {
  batch_wait(q);
  int fl = __builtin_amdgcn_readfirstlane(q.f.x | q.f.y | q.f.z | q.f.w);
  if (__builtin_amdgcn_readfirstlane(min(min(q.f.x, q.f.y), min(q.f.z, q.f.w))) == 0) {
    fl = wait_group(fbase + T * 16, t0);
    batch_issue<T>(q, fbase, e0, e1);
    batch_wait(q);
  }
  return (fl & 2) ? __builtin_nanf("") : sc0;
}
template <int T> __device__ __forceinline__ void batch_store(const Batch& q, unsigned ovoff, float* o0, float* o1, float sc) {
  const float k1 = -sc * 1.4426950408889634f;
  v4f r0 = silu4((q.p[0] + q.p[1]) + (q.p[2] + q.p[3]), sc, k1);
  v4f r1 = silu4((q.p[4] + q.p[5]) + (q.p[6] + q.p[7]), sc, k1);
  asm volatile("global_store_dwordx4 %0, %1, %3 offset:%c5 sc1\n\tglobal_store_dwordx4 %0, %2, %4 offset:%c5 sc1\n\ts_nop 1"
               :: "v"(ovoff), "v"(r0), "v"(r1), "s"(o0), "s"(o1), "i"(T * 128) : "memory");
}

__global__ __launch_bounds__(512) void a2a_gemm_kernel(Args a) {
  extern __shared__ __attribute__((aligned(16))) uint8_t smem[];
  const int b = blockIdx.x;
  const int lane = threadIdx.x & 63;
  const int w = __builtin_amdgcn_readfirstlane(threadIdx.x >> 6);
  const int xcd = b & 7, idx = b >> 3;
  const int ntile = ((xcd + 2 * (idx & 3)) & 7) + 8 * (idx & 3);
  const int mtile = idx >> 2;
  const int rank = a.rank;
  uint8_t* const myws = a.myws;
  unsigned* const ctr = (unsigned*)(myws + WS_CTR + (size_t)b * 128);
  unsigned call; float sxv, swv;
  asm volatile("" :: "s"(a.x), "s"(a.W), "s"(a.sx), "s"(a.sw), "s"(a.out), "s"(myws), "s"(a.slow), "s"(rank));
#define LOAD_SCALARS() asm volatile("s_load_dword %0, %3, 0x0\n\ts_load_dword %1, %4, 0x0\n\ts_load_dword %2, %5, 0x0\n\ts_waitcnt lgkmcnt(0)" \
               : "=&s"(call), "=&s"(sxv), "=&s"(swv) : "s"(ctr), "s"(a.sx), "s"(a.sw) : "memory")
  const unsigned long long t0 = __builtin_amdgcn_s_memrealtime();
  const unsigned fbase = (unsigned)(uintptr_t)smem + LDS_FLAGS;
  const unsigned ovoff = (unsigned)((((lane >> 3) & 1) * 4 + (lane >> 4)) * 16384 + (lane & 7) * 16);
  float* const o0 = a.out + (size_t)(mtile * 64 + (w & 3) * 8) * 4096 + ntile * 128;
  float* const o1 = o0 + 32 * 4096;
  const unsigned e0 = (unsigned)(uintptr_t)smem + (w & 3) * 1024 + lane * 16, e1 = e0 + 65536;

  if (w < 4) {
    __builtin_amdgcn_s_setprio(3);
    const int pl = JMAP(mtile), pm = ntile >> 2, pr = ntile & 3;
    const int pj = (pl + ROTM * pm + ROTS * rank) & 7;
    const unsigned d0 = (unsigned)(rank + 1 + w) & 7u, d1 = (unsigned)(rank + 5 + w) & 7u;
    const unsigned stage = (unsigned)(uintptr_t)smem + LDS_STAGE + w * 2048;
    const uint8_t* src = a.x + (size_t)(pm * 64 + pr * 16) * 512 + pj * 64;
    const unsigned sv = (unsigned)((lane >> 2) * 512 + (lane & 3) * 16);
    asm volatile("s_mov_b32 m0, %0\n\ts_nop 0\n\tglobal_load_lds_dwordx4 %1, %2" :: "s"(stage), "v"(sv), "s"(src + (size_t)d0 * (512 * 512)) : "memory");
    if (w < 3) asm volatile("s_mov_b32 m0, %0\n\ts_nop 0\n\tglobal_load_lds_dwordx4 %1, %2" :: "s"(stage + 1024), "v"(sv), "s"(src + (size_t)d1 * (512 * 512)) : "memory");
    if (w == 0) asm volatile("ds_write_b32 %0, %1\n\ts_waitcnt lgkmcnt(0)" :: "v"(fbase + (lane & 31) * 4), "v"(0) : "memory");
    __builtin_amdgcn_s_barrier();
    uint8_t* const ws0 = a.peer[w];
    uint8_t* const ws1 = a.peer[w + 4];
    LOAD_SCALARS();
    asm volatile("ds_write_b32 %0, %1 offset:64" :: "v"(fbase + w * 4), "v"(1) : "memory");
    const size_t doff = WS_XBUF + (size_t)(call % NBUF) * XBUF_BYTES + (size_t)(rank * 64 + pj * 8 + pm) * 4096 + (pr >> 1) * 2048 + (pr & 1) * 256;
    const unsigned so = (unsigned)(((lane >> 4) & 1) * 1024 + (lane >> 5) * 512 + (lane & 15) * 16);
    const unsigned ra = stage + (lane & 15) * 64 + (lane >> 4) * 16;
    v4i v0, v1;
    asm volatile("s_waitcnt vmcnt(0)\n\tds_read_b128 %0, %2\n\tds_read_b128 %1, %2 offset:1024\n\ts_waitcnt lgkmcnt(0)"
                 : "=&v"(v0), "=&v"(v1) : "v"(ra) : "memory");
    v0.x = v0.x ? v0.x : 0x80; v0.y = v0.y ? v0.y : 0x80; v0.z = v0.z ? v0.z : 0x80; v0.w = v0.w ? v0.w : 0x80;
    v1.x = v1.x ? v1.x : 0x80; v1.y = v1.y ? v1.y : 0x80; v1.z = v1.z ? v1.z : 0x80; v1.w = v1.w ? v1.w : 0x80;
    const int slot = pl * 32 + ntile - SLOT0;
    if (slot > 0) {
      const unsigned long long due = t0 + TBASE + (((unsigned)slot * SLOT8) >> 3);
      while (__builtin_amdgcn_s_memrealtime() < due) __builtin_amdgcn_s_sleep(1);
    }
    asm volatile("global_store_dwordx4 %0, %1, %2 sc0 sc1\n\ts_nop 1" :: "v"(so), "v"(v0), "s"(ws0 + doff) : "memory");
    if (w < 3) asm volatile("global_store_dwordx4 %0, %1, %2 sc0 sc1\n\ts_nop 1" :: "v"(so), "v"(v1), "s"(ws1 + doff) : "memory");
    __builtin_amdgcn_s_setprio(0);
#pragma unroll
    for (int i = 0; i < 14; i++) if ((BMASK >> i) & 1) __builtin_amdgcn_s_barrier();
    for (;;) {
      unsigned v;
      asm volatile("ds_read_b32 %0, %1 offset:80\n\ts_waitcnt lgkmcnt(0)" : "=v"(v) : "v"(fbase) : "memory");
      if (__builtin_amdgcn_readfirstlane(v)) break;
      if (__builtin_amdgcn_s_memrealtime() - t0 > 2 * DEADLINE_TICKS) break;
      __builtin_amdgcn_s_sleep(4);
    }
    {
      uint8_t* z = myws + WS_XBUF + (size_t)((call + 2u) % NBUF) * XBUF_BYTES + (size_t)b * 1024;
      const v4i zero = {0, 0, 0, 0};
      const unsigned zo = lane * 16;
      asm volatile("global_store_dwordx4 %0, %1, %2 sc1\n\ts_nop 1" :: "v"(zo), "v"(zero), "s"(z + (size_t)d0 * 262144) : "memory");
      if (w < 3) asm volatile("global_store_dwordx4 %0, %1, %2 sc1\n\ts_nop 1" :: "v"(zo), "v"(zero), "s"(z + (size_t)d1 * 262144) : "memory");
    }
#define HS(i)
    const float sc0 = sxv * swv;
    if (w == 0) (void)wait_group(fbase + 64, t0);
    (void)wait_group(fbase, t0);
    HS(20);
    Batch qa, qb;
    batch_issue<0>(qa, fbase, e0, e1);
    if (w == 0 && lane == 0) __hip_atomic_store(ctr, call + 1u, __ATOMIC_RELAXED, __HIP_MEMORY_SCOPE_AGENT);
    float sc = batch_finish<0>(qa, fbase, e0, e1, t0, sc0);
    HS(3);
    batch_issue<1>(qb, fbase, e0, e1);
    batch_store<0>(qa, ovoff, o0, o1, sc);
    HS(4);
    sc = batch_finish<1>(qb, fbase, e0, e1, t0, sc0);
    HS(5);
    batch_issue<2>(qa, fbase, e0, e1);
    batch_store<1>(qb, ovoff, o0, o1, sc);
    HS(6);
    sc = batch_finish<2>(qa, fbase, e0, e1, t0, sc0);
    HS(7);
    batch_store<2>(qa, ovoff, o0, o1, sc);
    HS(8);
  } else {
    CW c;
    __builtin_amdgcn_s_setprio(CPRIO);
    c.cw = w - 4; c.rank = rank; c.mtile = mtile; c.mode = 0; c.dead = 0;
    c.t0 = t0;
    c.lds0 = (unsigned)(uintptr_t)smem + c.cw * 32768;
    c.faddr = fbase + c.cw * 4;
    c.wv0 = (unsigned)((lane >> 3) * 4096 + (((lane & 7) ^ (((lane >> 4) & 3) << 1)) << 4));
    c.wv1 = c.wv0 + 8 * 4096 - 1024;
    c.Wn = a.W + ntile * 128;
    c.av = lane * 16;
    {
      const int dlo = (a.slow - xcd) & 7;
      if ((dlo & 1) == 0 && c.cw < 2)
        asm volatile("s_mov_b32 m0, %0\n\ts_nop 0\n\tglobal_load_lds_dword %1, %2"
                     :: "s"((unsigned)(uintptr_t)smem + LDS_JUNK), "v"(c.av << 8),
                        "s"(a.W + (a.slow + 8 * (dlo >> 1)) * 128 + (size_t)(idx * 128 + c.cw * 64) * 4096) : "memory");
    }
    c.pair<0>(); c.pair<1>(); c.pair<2>(); c.pair<3>();
    __builtin_amdgcn_s_barrier();
    {
      int g = lane >> 4, h = g >> 1, cg = g & 1, i = lane & 15, q = i >> 1, p = i & 1;
      int sw = ((q >> 1) & 3) << 1;
#pragma unroll
      for (int t = 0; t < 4; t++) c.baddr[t] = c.lds0 + (32 * h + q) * 128 + (((2 * t + cg) ^ sw) << 4) + 8 * p;
    }
#pragma unroll
    for (int i = 0; i < 2; i++)
#pragma unroll
      for (int t = 0; t < 4; t++) c.acc[i][t] = (v16f){};
    const unsigned stage = c.lds0 + 3 * 8192;
    {
      const unsigned xve = (unsigned)((lane >> 3) * 512 + (((lane & 7) ^ ((lane >> 4) & 3)) << 4));
      const unsigned xvo = (xve ^ 64u) + 8 * 512 - 1024;
      const uint8_t* xo = a.x + (size_t)(rank * 512 + mtile * 64) * 512 + c.cw * 128;
      dma2(stage, xve, xvo, xo); dma2(stage + 2048, xve, xvo, xo + 16 * 512);
      dma2(stage + 4096, xve, xvo, xo + 32 * 512); dma2(stage + 6144, xve, xvo, xo + 48 * 512);
    }
    c.pair<4>(); c.pair<5>(); c.pair<6>(); c.pair<7>();
    if constexpr (WEND[0] > 8) { c.pair<8>(); } if constexpr (WEND[0] > 9) { c.pair<9>(); }
    if constexpr (WEND[0] > 10) { c.pair<10>(); } if constexpr (WEND[0] > 11) { c.pair<11>(); }
    LOAD_SCALARS();
    c.xb = myws + WS_XBUF + (size_t)(call % NBUF) * XBUF_BYTES + (size_t)mtile * 4096;
    asm volatile("s_waitcnt vmcnt(%c0)" :: "i"(SC.tot[0] - SC.posA[0]) : "memory");
    {
      const unsigned xr0 = stage + (unsigned)((lane & 31) * 128), xs = (unsigned)(((2 * (lane >> 5)) ^ ((lane >> 1) & 7)) << 4);
      readX(c.A[0], xr0 + xs, xr0 + (xs ^ 16u));
      readX(c.A[1], xr0 + (xs ^ 64u), xr0 + (xs ^ 80u));
      waitX(c.A[0], c.A[1]);
    }
    readB<0>(c.B[0], c.baddr[0]); readB<0>(c.B[1], c.baddr[1]); readB<0>(c.B[2], c.baddr[2]); readB<0>(c.B[3], c.baddr[3]);
    c.step<0>(); c.step<1>(); c.step<2>(); c.step<3>(); c.step<4>(); c.step<5>(); c.step<6>();
    c.step<7>(); c.step<8>(); c.step<9>(); c.step<10>(); c.step<11>(); c.step<12>();
    c.tail();
    Batch q;
    batch_issue<3>(q, fbase, e0, e1);
    const float sc = batch_finish<3>(q, fbase, e0, e1, t0, sxv * swv);
    HS(20);
    batch_store<3>(q, ovoff, o0, o1, sc);
  }
}

extern "C" void kernel_launch(int rank, int nranks, ncclComm_t comm, hipStream_t stream,
                              void* const* d_in, const int* in_sizes, int n_in,
                              void* d_out, int out_size, const int* in_dtypes, int out_dtype,
                              void* const* d_out_all, void* const* d_ws_all, size_t ws_bytes) {
  if (nranks != NRANKS || ws_bytes < WS_NEED || (unsigned)rank >= (unsigned)NRANKS) {
    fprintf(stderr, "unsupported configuration\n");
    return;
  }
  static bool init = false;
  if (!init) {
    HIP_CHECK(hipFuncSetAttribute((const void*)a2a_gemm_kernel, hipFuncAttributeMaxDynamicSharedMemorySize, LDS_BYTES));
    init = true;
  }
  Args a;
  a.x = (const uint8_t*)d_in[0]; a.W = (const uint8_t*)d_in[1];
  a.sx = (const float*)d_in[2]; a.sw = (const float*)d_in[3];
  a.out = (float*)d_out;
  a.myws = (uint8_t*)d_ws_all[rank];
  for (int p = 0; p < NRANKS; p++) a.peer[p] = (uint8_t*)d_ws_all[(rank + 1 + p) % NRANKS];
  a.rank = rank; a.slow = (int)((3u - (unsigned)((uintptr_t)d_in[1] >> 7)) & 7u);
  a2a_gemm_kernel<<<GRID, 512, LDS_BYTES, stream>>>(a);
}
```

```cpp
#include <hip/hip_runtime.h>
#include <rccl/rccl.h>
#include <cstdio>
#include <cstdint>

#define HIP_CHECK(expr) do { hipError_t _e = (expr); if (_e != hipSuccess) { \
    fprintf(stderr, "HIP error %s at %s:%d\n", hipGetErrorString(_e), __FILE__, __LINE__); \
    return; } } while (0)

typedef int v2i __attribute__((ext_vector_type(2)));
typedef int v4i __attribute__((ext_vector_type(4)));
typedef int v8i __attribute__((ext_vector_type(8)));
typedef float v4f __attribute__((ext_vector_type(4)));
typedef float v16f __attribute__((ext_vector_type(16)));

constexpr int NRANKS = 8;
constexpr int GRID = 256;
constexpr int SLOT8 = 19, SLOT0 = 24, TBASE = 60;
constexpr int BSTEP = 12;

constexpr int LDS_STAGE = 131072;
constexpr int LDS_FLAGS = LDS_STAGE + 8192;
constexpr int LDS_JUNK = LDS_FLAGS + 128;
constexpr int LDS_BYTES = LDS_JUNK + 256;

constexpr size_t WS_CTR = 0;
constexpr size_t WS_TMO = 64 << 10;
constexpr size_t WS_XBUF = 2 << 20;
constexpr unsigned NBUF = 4;
constexpr unsigned XBUF_BYTES = 256 << 10;
constexpr unsigned XSRC_BYTES = NBUF * XBUF_BYTES;
constexpr size_t WS_NEED = WS_XBUF + (size_t)NRANKS * XSRC_BYTES;
__device__ __forceinline__ unsigned xslot(unsigned src, unsigned call) { return src * XSRC_BYTES + (call % NBUF) * XBUF_BYTES; }
constexpr unsigned long long DEADLINE_TICKS = 50000000ull;

struct Args {
  const uint8_t* x; const uint8_t* W; const float* sx; const float* sw; float* out;
  uint8_t* myws;
  int rank, slow;
  uint8_t* peer[NRANKS];
};

constexpr int WEND[14] = {12, 20, 24, 28, 32, 36, 40, 44, 48, 52, 56, 60, 64, 64};
constexpr int AST[16] = {0, 0, 1, 1, 2, 3, 4, 5, 6, 7, 8, 9, 10, 11, 12, 12};
constexpr int AGP[16] = {0, 0, 3, 3, -1, -1, -1, -1, -1, -1, -1, -1, -1, -1, 0, 2};
constexpr int gap_lo(int s, int g) {
  int n = WEND[s + 1] - WEND[s];
  return WEND[s] + (n * g + 5) / 6;
}
constexpr int cmin(int a, int b) { return a < b ? a : b; }
constexpr int cmax(int a, int b) { return a > b ? a : b; }
struct Sched {
  int posA[16], posW[16], tot[14], mid[8];
  constexpr Sched() : posA(), posW(), tot(), mid() {
    int n = 0;
    n += 8; posW[0] = n;
    n += 8; posA[0] = posA[1] = n;
    for (int p = 4; p < WEND[0]; p++) { n += 2; if ((p & 3) == 3) posW[p >> 2] = n; }
    for (int s = 0; s < 13; s++) {
      tot[s] = n;
      for (int g = -1; g <= 6; g++) {
        if (s == 12 && g >= 0) mid[g] = n;
        for (int a = 2; a < 16; a++) if (AST[a] == s && AGP[a] == g) { n += 4; posA[a] = n; }
        if (g >= 0 && g < 6) for (int p = gap_lo(s, g); p < gap_lo(s, g + 1); p++) { n += 2; if ((p & 3) == 3) posW[p >> 2] = n; }
      }
    }
    tot[13] = n;
  }
};
constexpr Sched SC;
constexpr int need_normal(int s) { return s >= 13 ? SC.posA[s] : cmax(SC.posA[s], SC.posW[s + 1]); }
constexpr int vm_normal(int s) { return cmin(63, SC.tot[cmin(s, 13)] - need_normal(s)); }
constexpr bool can_look(int s) { return s < 15 && AST[s + 1] < s; }
constexpr int vm_mode(int s) {
  return (s < 3 || s > 13) ? 0 : cmin(63, SC.tot[s] - cmax(need_normal(s), cmax(SC.tot[s - 1], can_look(s) ? SC.posA[s + 1] : 0)));
}
constexpr bool sched_ok() {
  if (WEND[0] != 12 || WEND[13] != 64) return false;
  for (int s = 0; s < 13; s++) {
    if (WEND[s + 1] < WEND[s] || WEND[s + 1] - WEND[s] > 8) return false;
    if (WEND[s + 1] > 4 * (s + 5)) return false;
    if (WEND[s] < 4 * (s + 2)) return false;
    if (vm_normal(s) < 0 || vm_mode(s) < 0) return false;
  }
  if (SC.mid[4] < SC.posW[14] || SC.mid[6] < SC.posW[15]) return false;
  for (int a = 2; a < 16; a++) {
    if (AST[a] >= cmin(a, 13) || AST[a] < a - 3 || AST[a] < 0 || AGP[a] > 5) return false;
    if (a > 2 && SC.posA[a] < SC.posA[a - 1]) return false;
  }
  return true;
}
static_assert(sched_ok(), "bad schedule");
static_assert(BSTEP >= 2 && BSTEP <= 12, "the barrier precedes every flag write");

__device__ __forceinline__ v8i cat(v4i a, v4i b) { return __builtin_shufflevector(a, b, 0, 1, 2, 3, 4, 5, 6, 7); }
__device__ __forceinline__ v8i cat4(const v2i (&b)[4]) {
  v4i x = __builtin_shufflevector(b[0], b[1], 0, 1, 2, 3), y = __builtin_shufflevector(b[2], b[3], 0, 1, 2, 3);
  return cat(x, y);
}
__device__ __forceinline__ void loadA(v4i (&a)[4], unsigned voff, const uint8_t* base) {
  asm volatile("global_load_dwordx4 %0, %4, %5 sc0 sc1\n\t"
               "global_load_dwordx4 %1, %4, %5 offset:1024 sc0 sc1\n\t"
               "global_load_dwordx4 %2, %4, %5 offset:2048 sc0 sc1\n\t"
               "global_load_dwordx4 %3, %4, %5 offset:3072 sc0 sc1"
               : "=&v"(a[0]), "=&v"(a[1]), "=&v"(a[2]), "=&v"(a[3]) : "v"(voff), "s"(base) : "memory");
}
__device__ __forceinline__ void dma2(unsigned m0v, unsigned voff0, unsigned voff1, const uint8_t* base) {
  asm volatile("s_mov_b32 m0, %0\n\ts_nop 0\n\t"
               "global_load_lds_dwordx4 %1, %3\n\t"
               "global_load_lds_dwordx4 %2, %3 offset:1024"
               :: "s"(m0v), "v"(voff0), "v"(voff1), "s"(base) : "memory");
}
__device__ __forceinline__ void dma1(unsigned m0v, unsigned voff, const uint8_t* base) {
  asm volatile("s_mov_b32 m0, %0\n\ts_nop 0\n\tglobal_load_lds_dwordx4 %1, %2" :: "s"(m0v), "v"(voff), "s"(base) : "memory");
}
__device__ __forceinline__ void readX(v4i (&a)[4], unsigned a0, unsigned a1) {
  asm volatile("ds_read_b128 %0, %4\n\tds_read_b128 %1, %5\n\tds_read_b128 %2, %4 offset:4096\n\tds_read_b128 %3, %5 offset:4096"
               : "=&v"(a[0]), "=&v"(a[1]), "=&v"(a[2]), "=&v"(a[3]) : "v"(a0), "v"(a1) : "memory");
}
__device__ __forceinline__ void waitX(v4i (&a)[4], v4i (&b)[4]) {
  asm volatile("s_waitcnt lgkmcnt(0)" : "+v"(a[0]), "+v"(a[1]), "+v"(a[2]), "+v"(a[3]), "+v"(b[0]), "+v"(b[1]), "+v"(b[2]), "+v"(b[3]) :: "memory");
}
template <int OFF, int R> __device__ __forceinline__ void dump8(const v16f& c) {
  asm volatile("ds_write_addtid_b32 %0 offset:%c8\n\tds_write_addtid_b32 %1 offset:%c9\n\tds_write_addtid_b32 %2 offset:%c10\n\tds_write_addtid_b32 %3 offset:%c11\n\t"
               "ds_write_addtid_b32 %4 offset:%c12\n\tds_write_addtid_b32 %5 offset:%c13\n\tds_write_addtid_b32 %6 offset:%c14\n\tds_write_addtid_b32 %7 offset:%c15"
               :: "v"(c[R]), "v"(c[R + 1]), "v"(c[R + 2]), "v"(c[R + 3]), "v"(c[R + 4]), "v"(c[R + 5]), "v"(c[R + 6]), "v"(c[R + 7]),
                  "i"(OFF), "i"(OFF + 256), "i"(OFF + 512), "i"(OFF + 768), "i"(OFF + 1024), "i"(OFF + 1280), "i"(OFF + 1536), "i"(OFF + 1792) : "memory");
}
template <int OFF> __device__ __forceinline__ void readB(v2i (&b)[4], unsigned addr) {
  asm volatile("ds_read_b64_tr_b8 %0, %4 offset:%c5\n\t"
               "ds_read_b64_tr_b8 %1, %4 offset:%c6\n\t"
               "ds_read_b64_tr_b8 %2, %4 offset:%c7\n\t"
               "ds_read_b64_tr_b8 %3, %4 offset:%c8"
               : "=&v"(b[0]), "=&v"(b[1]), "=&v"(b[2]), "=&v"(b[3])
               : "v"(addr), "i"(OFF), "i"(OFF + 1024), "i"(OFF + 2048), "i"(OFF + 3072) : "memory");
}
template <int N> __device__ __forceinline__ void waitA(v4i (&a)[4]) {
  asm volatile("s_waitcnt vmcnt(%c4)" : "+v"(a[0]), "+v"(a[1]), "+v"(a[2]), "+v"(a[3]) : "i"(N) : "memory");
}
template <int N> __device__ __forceinline__ void waitAA(v4i (&a)[4], v4i (&b)[4]) {
  asm volatile("s_waitcnt vmcnt(%c8)" : "+v"(a[0]), "+v"(a[1]), "+v"(a[2]), "+v"(a[3]), "+v"(b[0]), "+v"(b[1]), "+v"(b[2]), "+v"(b[3]) : "i"(N) : "memory");
}
template <int N0, int N1> __device__ __forceinline__ void waitA1m(v4i (&a)[4], int mode) {
  asm volatile("s_cmp_eq_u32 %4, 0\n\ts_cbranch_scc1 1f\n\ts_waitcnt vmcnt(%c6)\n1:\n\ts_waitcnt vmcnt(%c5)"
               : "+v"(a[0]), "+v"(a[1]), "+v"(a[2]), "+v"(a[3]) : "s"(mode), "i"(N0), "i"(N1) : "memory", "scc");
}
template <int N0, int N1> __device__ __forceinline__ void waitA2m(v4i (&a)[4], v4i (&b)[4], int mode) {
  asm volatile("s_cmp_eq_u32 %8, 0\n\ts_cbranch_scc1 1f\n\ts_waitcnt vmcnt(%c10)\n1:\n\ts_waitcnt vmcnt(%c9)"
               : "+v"(a[0]), "+v"(a[1]), "+v"(a[2]), "+v"(a[3]), "+v"(b[0]), "+v"(b[1]), "+v"(b[2]), "+v"(b[3])
               : "s"(mode), "i"(N0), "i"(N1) : "memory", "scc");
}
template <int N> __device__ __forceinline__ void waitB(v2i (&b)[4]) {
  asm volatile("s_waitcnt lgkmcnt(%c4)" : "+v"(b[0]), "+v"(b[1]), "+v"(b[2]), "+v"(b[3]) : "i"(N) : "memory");
}
template <int N> __device__ __forceinline__ void waitB3(v2i (&b)[4], v2i (&c)[4], v2i (&d)[4]) {
  asm volatile("s_waitcnt lgkmcnt(%c12)" : "+v"(b[0]), "+v"(b[1]), "+v"(b[2]), "+v"(b[3]), "+v"(c[0]), "+v"(c[1]), "+v"(c[2]), "+v"(c[3]),
               "+v"(d[0]), "+v"(d[1]), "+v"(d[2]), "+v"(d[3]) : "i"(N) : "memory");
}
__device__ __forceinline__ unsigned umin3(unsigned a, unsigned b, unsigned c) { return min(min(a, b), c); }
__device__ __forceinline__ bool missing(const v4i (&a)[4]) {
  unsigned m0 = umin3(a[0].x, a[0].y, a[0].z), m1 = umin3(a[0].w, a[1].x, a[1].y);
  m0 = umin3(m0, a[1].z, a[1].w); m1 = umin3(m1, a[2].x, a[2].y);
  m0 = umin3(m0, a[2].z, a[2].w); m1 = umin3(m1, a[3].x, a[3].y);
  m0 = umin3(m0, a[3].z, a[3].w);
  return __builtin_amdgcn_ballot_w64(min(m0, m1) == 0u) != 0ull;
}
__device__ __forceinline__ v4i nonzero(v4i v) {
  v.x = v.x ? v.x : 0x80; v.y = v.y ? v.y : 0x80; v.z = v.z ? v.z : 0x80; v.w = v.w ? v.w : 0x80;
  return v;
}

__device__ __forceinline__ int slice_of(int s, int cw, int rank, int mtile) {
  if (s < 2) return rank * 8 + 2 * cw + s;
  int q = (s - 2) * 4 + cw;
  int lv = (q * 37) >> 8;
  int src = (rank + 1 + q - lv * 7) & 7;
  return src * 8 + ((lv + mtile + src) & 7);
}

struct CW {
  v16f acc[2][4];
  v4i A[4][4];
  v2i B[4][4];
  unsigned baddr[4], wv0, wv1, av, lds0, faddr;
  const uint8_t* Wn; const uint8_t* xb;
  int cw, rank, mode, dead, mtile;
  unsigned call;
  unsigned long long t0;
  __device__ __forceinline__ const uint8_t* wbase(int s) const { return Wn + (size_t)slice_of(s, cw, rank, mtile) * (64 * 4096); }
  __device__ __forceinline__ const uint8_t* abase(int s) const {
    const unsigned sl = (unsigned)slice_of(s, cw, rank, mtile);
    return xb + (size_t)(xslot(sl >> 3, call) + (sl & 7) * (8 * 4096));
  }
  template <int P> __device__ __forceinline__ void pair() {
    dma2(lds0 + ((P >> 2) & 3) * 8192 + (P & 3) * 2048, wv0, wv1, wbase(P >> 2) + (P & 3) * (16 * 4096));
  }
  template <int S, int G, int AI> __device__ __forceinline__ void aissue() {
    if constexpr (AI < 16) {
      if constexpr (AST[AI] == S && AGP[AI] == G) loadA(A[AI & 3], av, abase(AI));
      aissue<S, G, AI + 1>();
    }
  }
  template <int S, int G> __device__ __forceinline__ void gap() {
    aissue<S, G, 2>();
    constexpr int lo = gap_lo(S, G), hi = gap_lo(S, G + 1);
    if constexpr (hi - lo >= 1) pair<lo>();
    if constexpr (hi - lo >= 2) pair<lo + 1>();
    static_assert(hi - lo <= 2, "");
  }
  template <int S, int I, int T, int F> __device__ __forceinline__ void mf() {
    constexpr int U = S & 3;
    acc[I][T] = __builtin_amdgcn_mfma_scale_f32_32x32x64_f8f6f4(cat(A[U][2 * I], A[U][2 * I + 1]), cat4(B[F]), acc[I][T], 0, 0, 0, 0, 0, 0);
    __builtin_amdgcn_sched_barrier(0);
  }
  template <int S, int N0, int N1, bool LOOK> __device__ __forceinline__ void ensure() {
    constexpr int U = S & 3, U1 = (S + 1) & 3;
    if constexpr (LOOK) waitA2m<N0, N1>(A[U], A[U1], mode); else waitA1m<N0, N1>(A[U], mode);
    int look = mode;
    if (__builtin_expect(missing(A[U]), 0)) {
      do {
        if (dead) break;
        __builtin_amdgcn_s_sleep(2);
        loadA(A[U], av, abase(S));
        if constexpr (LOOK) waitAA<0>(A[U], A[U1]); else waitA<0>(A[U]);
        if (__builtin_amdgcn_s_memrealtime() - t0 > DEADLINE_TICKS) dead = 1;
      } while (missing(A[U]));
      look = dead ? 0 : 1;
    }
    mode = 0;
    if constexpr (LOOK) {
      if (__builtin_expect(look, 0)) {
        if (missing(A[U1])) { loadA(A[U1], av, abase(S + 1)); mode = 1; }
      }
    }
  }
  template <int S> __device__ __forceinline__ void step() {
    constexpr int BOFF = ((S + 1) & 3) * 8192;
    if constexpr (S < 2) waitA<vm_normal(S)>(A[S]);
    else ensure<S, vm_normal(S), vm_mode(S), can_look(S)>();
    if constexpr (S == BSTEP) __builtin_amdgcn_s_barrier();
    aissue<S, -1, 2>();
    __builtin_amdgcn_sched_barrier(0);
    waitB<12>(B[0]);
    mf<S, 0, 0, 0>(); mf<S, 1, 0, 0>();
    readB<BOFF>(B[0], baddr[0]);
    waitB3<4>(B[1], B[2], B[3]);
    gap<S, 0>();
    mf<S, 0, 1, 1>();
    gap<S, 1>();
    mf<S, 1, 1, 1>();
    readB<BOFF>(B[1], baddr[1]);
    gap<S, 2>();
    mf<S, 0, 2, 2>();
    gap<S, 3>();
    mf<S, 1, 2, 2>();
    if constexpr (S < 12) readB<BOFF>(B[2], baddr[2]);
    else { asm volatile("s_waitcnt vmcnt(%c0)" :: "i"(cmin(63, SC.mid[4] - SC.posW[14])) : "memory"); readB<2 * 8192>(B[2], baddr[0]); }
    gap<S, 4>();
    mf<S, 0, 3, 3>();
    gap<S, 5>();
    mf<S, 1, 3, 3>();
    if constexpr (S < 12) readB<BOFF>(B[3], baddr[3]);
    else { asm volatile("s_waitcnt vmcnt(%c0)" :: "i"(cmin(63, SC.mid[6] - SC.posW[15])) : "memory"); readB<3 * 8192>(B[3], baddr[0]); }
    __builtin_amdgcn_sched_barrier(0);
  }
  template <int T> __device__ __forceinline__ void flag() {
    asm volatile("ds_write_b32 %0, %1 offset:%c2" :: "v"(faddr), "v"(1 + dead), "i"(T * 16) : "memory");
    __builtin_amdgcn_sched_barrier(0);
  }
  template <int T, int Q> __device__ __forceinline__ void dump() {
    dump8<T * 8192 + Q * 2048, (Q & 1) * 8>(acc[Q >> 1][T]);
    __builtin_amdgcn_sched_barrier(0);
  }
  __device__ __forceinline__ void tail() {
    ensure<13, vm_normal(13), vm_mode(13), can_look(13)>();
    asm volatile("s_mov_b32 m0, %0" :: "s"(lds0) : "memory");
    __builtin_amdgcn_sched_barrier(0);
    waitB<12>(B[0]);
    mf<13, 0, 0, 0>(); mf<13, 1, 0, 0>();
    readB<2 * 8192>(B[0], baddr[1]);
    waitB<12>(B[1]);
    mf<13, 0, 1, 1>(); mf<13, 1, 1, 1>();
    readB<3 * 8192>(B[1], baddr[1]);
    ensure<14, vm_normal(14), 0, true>();
    waitB<12>(B[2]);
    mf<14, 0, 0, 2>(); mf<14, 1, 0, 2>();
    readB<1 * 8192>(B[2], baddr[2]);
    ensure<15, 0, 0, false>();
    waitB<12>(B[3]);
    mf<15, 0, 0, 3>(); mf<15, 1, 0, 3>();
    readB<2 * 8192>(B[3], baddr[2]);
    waitB<8>(B[0]); waitB<8>(B[1]);
    mf<14, 0, 1, 0>(); dump<0, 0>();
    mf<14, 1, 1, 0>();
    readB<3 * 8192>(B[0], baddr[2]);
    dump<0, 1>();
    mf<15, 0, 1, 1>(); dump<0, 2>();
    mf<15, 1, 1, 1>();
    readB<1 * 8192>(B[1], baddr[3]);
    dump<0, 3>(); flag<0>();
    waitB<15>(B[2]); waitB<15>(B[3]);
    mf<13, 0, 2, 2>(); mf<13, 1, 2, 2>();
    waitB<9>(B[1]); waitB<9>(B[0]);
    readB<2 * 8192>(B[2], baddr[3]);
    dump<1, 0>();
    mf<14, 0, 2, 3>(); dump<1, 1>();
    mf<14, 1, 2, 3>();
    readB<3 * 8192>(B[3], baddr[3]);
    dump<1, 2>();
    mf<15, 0, 2, 0>(); dump<1, 3>(); flag<1>();
    mf<15, 1, 2, 0>();
    waitB<15>(B[2]); waitB<15>(B[3]);
    mf<13, 0, 3, 1>(); dump<2, 0>();
    mf<13, 1, 3, 1>(); dump<2, 1>();
    mf<14, 0, 3, 2>(); dump<2, 2>();
    mf<14, 1, 3, 2>(); dump<2, 3>(); flag<2>();
    mf<15, 0, 3, 3>(); mf<15, 1, 3, 3>();
    asm volatile("s_nop 7" ::: "memory");
    dump<3, 0>(); dump<3, 1>();
    asm volatile("s_nop 7" ::: "memory");
    dump<3, 2>(); dump<3, 3>(); flag<3>();
  }
};

__device__ __forceinline__ int wait_group(unsigned faddr, unsigned long long t0) {
  for (unsigned it = 1;; it++) {
    v4i f;
    asm volatile("ds_read_b128 %0, %1\n\ts_waitcnt lgkmcnt(0)" : "=v"(f) : "v"(faddr) : "memory");
    int mn = __builtin_amdgcn_readfirstlane(min(min(f.x, f.y), min(f.z, f.w)));
    if (mn) return __builtin_amdgcn_readfirstlane(f.x | f.y | f.z | f.w);
    __builtin_amdgcn_s_sleep(1);
    if ((it & 255) == 0 && __builtin_amdgcn_s_memrealtime() - t0 > 2 * DEADLINE_TICKS) return 2;
  }
}
__device__ __forceinline__ v4f silu4(v4f v, float sc, float k1) {
  const float lim = 60.0f * 1.4426950408889634f;
  v4f u = v * k1, y = v * sc, r;
#pragma unroll
  for (int k = 0; k < 4; k++) u[k] = __builtin_amdgcn_exp2f(__builtin_amdgcn_fmed3f(u[k], -lim, lim));
  u += 1.0f;
#pragma unroll
  for (int k = 0; k < 4; k++) r[k] = __builtin_amdgcn_rcpf(u[k]);
  return y * r;
}
struct Batch { v4i f; v4f p[8]; };
template <int T> __device__ __forceinline__ void batch_issue(Batch& q, unsigned fbase, unsigned e0, unsigned e1) {
  asm volatile("ds_read_b128 %0, %9 offset:%c12\n\t"
               "ds_read_b128 %1, %10 offset:%c13\n\tds_read_b128 %2, %10 offset:%c14\n\tds_read_b128 %3, %11 offset:%c13\n\tds_read_b128 %4, %11 offset:%c14\n\t"
               "ds_read_b128 %5, %10 offset:%c15\n\tds_read_b128 %6, %10 offset:%c16\n\tds_read_b128 %7, %11 offset:%c15\n\tds_read_b128 %8, %11 offset:%c16"
               : "=&v"(q.f), "=&v"(q.p[0]), "=&v"(q.p[1]), "=&v"(q.p[2]), "=&v"(q.p[3]), "=&v"(q.p[4]), "=&v"(q.p[5]), "=&v"(q.p[6]), "=&v"(q.p[7])
               : "v"(fbase), "v"(e0), "v"(e1), "i"(T * 16), "i"(T * 8192), "i"(T * 8192 + 32768), "i"(T * 8192 + 4096), "i"(T * 8192 + 4096 + 32768) : "memory");
}
__device__ __forceinline__ void batch_wait(Batch& q) {
  asm volatile("s_waitcnt lgkmcnt(0)" : "+v"(q.f), "+v"(q.p[0]), "+v"(q.p[1]), "+v"(q.p[2]), "+v"(q.p[3]), "+v"(q.p[4]), "+v"(q.p[5]), "+v"(q.p[6]), "+v"(q.p[7]) :: "memory");
}
template <int T> __device__ __forceinline__ float batch_finish(Batch& q, unsigned fbase, unsigned e0, unsigned e1, unsigned long long t0, float sc0) {
  batch_wait(q);
  int fl = __builtin_amdgcn_readfirstlane(q.f.x | q.f.y | q.f.z | q.f.w);
  if (__builtin_amdgcn_readfirstlane(min(min(q.f.x, q.f.y), min(q.f.z, q.f.w))) == 0) {
    fl = wait_group(fbase + T * 16, t0);
    batch_issue<T>(q, fbase, e0, e1);
    batch_wait(q);
  }
  return (fl & 2) ? __builtin_nanf("") : sc0;
}
template <int T> __device__ __forceinline__ void batch_store(const Batch& q, unsigned ovoff, float* o0, float* o1, float sc) {
  const float k1 = -sc * 1.4426950408889634f;
  v4f r0 = silu4((q.p[0] + q.p[1]) + (q.p[2] + q.p[3]), sc, k1);
  v4f r1 = silu4((q.p[4] + q.p[5]) + (q.p[6] + q.p[7]), sc, k1);
  asm volatile("global_store_dwordx4 %0, %1, %3 offset:%c5 sc1\n\tglobal_store_dwordx4 %0, %2, %4 offset:%c5 sc1\n\ts_nop 1"
               :: "v"(ovoff), "v"(r0), "v"(r1), "s"(o0), "s"(o1), "i"(T * 128) : "memory");
}

__global__ __launch_bounds__(512) void a2a_gemm_kernel(Args a) {
  extern __shared__ __attribute__((aligned(16))) uint8_t smem[];
  const uint8_t* pc0;
  unsigned codelen;
  asm volatile("s_getpc_b64 %0\n.La2a_code_begin:\n\ts_mov_b32 %1, .La2a_code_end-.La2a_code_begin" : "=s"(pc0), "=s"(codelen));
  const int b = blockIdx.x;
  const int lane = threadIdx.x & 63;
  const int w = __builtin_amdgcn_readfirstlane(threadIdx.x >> 6);
  uint8_t *ws0, *ws1;
  asm volatile("s_load_dwordx2 %0, %2, %3 offset:%c4\n\ts_load_dwordx2 %1, %2, %3 offset:%c5"
               : "=&s"(ws0), "=&s"(ws1)
               : "s"(__builtin_amdgcn_kernarg_segment_ptr()), "s"((w & 3) * 8), "i"(__builtin_offsetof(Args, peer)), "i"(__builtin_offsetof(Args, peer) + 32) : "memory");
  const int xcd = b & 7, idx = b >> 3;
  const int ntile = ((xcd + 2 * (idx & 3)) & 7) + 8 * (idx & 3);
  const int mtile = idx >> 2;
  const int rank = a.rank;
  uint8_t* const myws = a.myws;
  unsigned* const ctr = (unsigned*)(myws + WS_CTR + (size_t)b * 128);
  unsigned call;
  asm volatile("s_waitcnt lgkmcnt(0)" : "+s"(ws0), "+s"(ws1)
               : "s"(a.x), "s"(a.W), "s"(a.sx), "s"(a.sw), "s"(a.out), "s"(myws), "s"(a.slow), "s"(rank) : "memory");
  unsigned call_raw;
  asm volatile("s_load_dword %0, %1, 0x0" : "=s"(call_raw) : "s"(ctr) : "memory");
#define WAIT_CALL() asm volatile("s_waitcnt lgkmcnt(0)\n\ts_mov_b32 %0, %1" : "=s"(call) : "s"(call_raw) : "memory")
  const unsigned long long t0 = __builtin_amdgcn_s_memrealtime();
  const unsigned fbase = (unsigned)(uintptr_t)smem + LDS_FLAGS;
  const unsigned ovoff = (unsigned)((((lane >> 3) & 1) * 4 + (lane >> 4)) * 16384 + (lane & 7) * 16);
  float* const o0 = a.out + (size_t)(mtile * 64 + (w & 3) * 8) * 4096 + ntile * 128;
  float* const o1 = o0 + 32 * 4096;
  const unsigned e0 = (unsigned)(uintptr_t)smem + (w & 3) * 1024 + lane * 16, e1 = e0 + 65536;

  const int pl = 7 - mtile, pm = ntile >> 2, pr = ntile & 3;
  const int pj = (pl + pm + rank) & 7;
  const unsigned d0 = (unsigned)(rank + 1 + w) & 7u, d1 = (unsigned)(rank + 5 + w) & 7u;
  const unsigned hstage = (unsigned)(uintptr_t)smem + LDS_STAGE + w * 2048;
  CW c;
  c.cw = w & 3; c.rank = rank; c.mtile = mtile; c.mode = 0; c.dead = 0;
  c.t0 = t0;
  c.lds0 = (unsigned)(uintptr_t)smem + c.cw * 32768;
  c.faddr = fbase + c.cw * 4;
  c.wv0 = (unsigned)((lane >> 3) * 4096 + (((lane & 7) ^ (((lane >> 4) & 3) << 1)) << 4));
  c.wv1 = c.wv0 + 8 * 4096 - 1024;
  c.Wn = a.W + ntile * 128;
  c.av = lane * 16;
  const unsigned stage = c.lds0 + 3 * 8192;
  if (__builtin_expect(w < 4, 1)) {
    __builtin_amdgcn_s_setprio(3);
    const uint8_t* src = a.x + (size_t)(pm * 64 + pr * 16) * 512 + pj * 64;
    const unsigned sv = (unsigned)((lane >> 2) * 512 + (lane & 3) * 16);
    dma1(hstage, sv, src + (size_t)d0 * (512 * 512));
    dma1(hstage + 1024, sv, src + (size_t)(w < 3 ? d1 : d0) * (512 * 512));
  }
  int w1 = w;
  asm volatile("" : "+s"(w1));
  if (__builtin_expect(w1 >= 4, 1)) {
    {
      const int dlo = (a.slow - xcd) & 7;
      if ((dlo & 1) == 0 && c.cw < 2)
        asm volatile("s_mov_b32 m0, %0\n\ts_nop 0\n\tglobal_load_lds_dword %1, %2"
                     :: "s"((unsigned)(uintptr_t)smem + LDS_JUNK), "v"(c.av << 8),
                        "s"(a.W + (a.slow + 8 * (dlo >> 1)) * 128 + (size_t)(idx * 128 + c.cw * 64) * 4096) : "memory");
    }
    if (c.cw == 3 && (unsigned)((lane * 32 + idx) * 128) < min(codelen, 16384u))
      asm volatile("s_mov_b32 m0, %0\n\ts_nop 0\n\tglobal_load_lds_dword %1, %2"
                   :: "s"((unsigned)(uintptr_t)smem + LDS_JUNK), "v"((unsigned)((lane * 32 + idx) * 128)), "s"(pc0) : "memory");
    c.pair<0>(); c.pair<1>(); c.pair<2>(); c.pair<3>();
    const unsigned xve = (unsigned)((lane >> 3) * 512 + (((lane & 7) ^ ((lane >> 4) & 3)) << 4));
    const unsigned xvo = (xve ^ 64u) + 8 * 512 - 1024;
    const uint8_t* xo = a.x + (size_t)(rank * 512 + mtile * 64) * 512 + c.cw * 128;
    dma2(stage, xve, xvo, xo); dma2(stage + 2048, xve, xvo, xo + 16 * 512);
    dma2(stage + 4096, xve, xvo, xo + 32 * 512); dma2(stage + 6144, xve, xvo, xo + 48 * 512);
  }
  int w2 = w;
  asm volatile("" : "+s"(w2));

  if (__builtin_expect(w2 < 4, 1)) {
    if (w == 3) asm volatile("ds_write_b32 %0, %1" :: "v"(fbase + (lane & 15) * 4), "v"(0) : "memory");
    WAIT_CALL();
    const size_t doff = WS_XBUF + (size_t)xslot(rank, call) + (size_t)(pj * 8 + pm) * 4096 + (pr >> 1) * 2048 + (pr & 1) * 256;
    const unsigned so = (unsigned)(((lane >> 4) & 1) * 1024 + (lane >> 5) * 512 + (lane & 15) * 16);
    const unsigned ra = hstage + (lane & 15) * 64 + (lane >> 4) * 16;
    v4i v0, v1;
    asm volatile("s_waitcnt vmcnt(0)\n\tds_read_b128 %0, %2\n\tds_read_b128 %1, %2 offset:1024\n\ts_waitcnt lgkmcnt(0)"
                 : "=&v"(v0), "=&v"(v1) : "v"(ra) : "memory");
    v0 = nonzero(v0); v1 = nonzero(v1);
    const int slot = pl * 32 + ntile - SLOT0;
    if (slot > 0) {
      const unsigned long long due = t0 + TBASE + (((unsigned)slot * SLOT8) >> 3);
      while (__builtin_amdgcn_s_memrealtime() < due) __builtin_amdgcn_s_sleep(1);
    }
    asm volatile("global_store_dwordx4 %0, %1, %2 sc0 sc1\n\ts_nop 1" :: "v"(so), "v"(v0), "s"(ws0 + doff) : "memory");
    if (w < 3) asm volatile("global_store_dwordx4 %0, %1, %2 sc0 sc1\n\ts_nop 1" :: "v"(so), "v"(v1), "s"(ws1 + doff) : "memory");
    __builtin_amdgcn_s_setprio(0);
    float sxv, swv;
    asm volatile("s_load_dword %0, %2, 0x0\n\ts_load_dword %1, %3, 0x0\n\ts_waitcnt lgkmcnt(0)" : "=&s"(sxv), "=&s"(swv) : "s"(a.sx), "s"(a.sw) : "memory");
    const float sc0 = sxv * swv;
    if (w == 3) asm volatile("ds_write_b32 %0, %1 offset:64\n\ts_waitcnt lgkmcnt(0)" :: "v"(fbase), "v"(sc0) : "memory");
    __builtin_amdgcn_s_barrier();
    {
      uint8_t* z = myws + WS_XBUF + (size_t)b * 1024;
      const v4i zero = {0, 0, 0, 0};
      const unsigned zo = lane * 16;
      asm volatile("global_store_dwordx4 %0, %1, %2 sc1\n\ts_nop 1" :: "v"(zo), "v"(zero), "s"(z + xslot(d0, call + 2u)) : "memory");
      if (w < 3) asm volatile("global_store_dwordx4 %0, %1, %2 sc1\n\ts_nop 1" :: "v"(zo), "v"(zero), "s"(z + xslot(d1, call + 2u)) : "memory");
      else if (lane == 0) __hip_atomic_store(ctr, call + 1u, __ATOMIC_RELAXED, __HIP_MEMORY_SCOPE_AGENT);
    }
    (void)wait_group(fbase, t0);
    Batch qa, qb;
    batch_issue<0>(qa, fbase, e0, e1);
    float sc = batch_finish<0>(qa, fbase, e0, e1, t0, sc0);
    batch_issue<1>(qb, fbase, e0, e1);
    batch_store<0>(qa, ovoff, o0, o1, sc);
    sc = batch_finish<1>(qb, fbase, e0, e1, t0, sc0);
    batch_issue<2>(qa, fbase, e0, e1);
    batch_store<1>(qb, ovoff, o0, o1, sc);
    sc = batch_finish<2>(qa, fbase, e0, e1, t0, sc0);
    batch_store<2>(qa, ovoff, o0, o1, sc);
  } else {
    c.pair<4>(); c.pair<5>(); c.pair<6>(); c.pair<7>();
    c.pair<8>(); c.pair<9>(); c.pair<10>(); c.pair<11>();
    {
      int g = lane >> 4, h = g >> 1, cg = g & 1, i = lane & 15, q = i >> 1, p = i & 1;
      int sw = ((q >> 1) & 3) << 1;
#pragma unroll
      for (int t = 0; t < 4; t++) c.baddr[t] = c.lds0 + (32 * h + q) * 128 + (((2 * t + cg) ^ sw) << 4) + 8 * p;
    }
#pragma unroll
    for (int i = 0; i < 2; i++)
#pragma unroll
      for (int t = 0; t < 4; t++) c.acc[i][t] = (v16f){};
    WAIT_CALL();
    c.call = call;
    c.xb = myws + WS_XBUF + (size_t)mtile * 4096;
    asm volatile("s_waitcnt vmcnt(%c0)" :: "i"(SC.tot[0] - SC.posA[0]) : "memory");
    {
      const unsigned xr0 = stage + (unsigned)((lane & 31) * 128), xs = (unsigned)(((2 * (lane >> 5)) ^ ((lane >> 1) & 7)) << 4);
      readX(c.A[0], xr0 + xs, xr0 + (xs ^ 16u));
      readX(c.A[1], xr0 + (xs ^ 64u), xr0 + (xs ^ 80u));
      waitX(c.A[0], c.A[1]);
    }
    readB<0>(c.B[0], c.baddr[0]); readB<0>(c.B[1], c.baddr[1]); readB<0>(c.B[2], c.baddr[2]); readB<0>(c.B[3], c.baddr[3]);
    c.step<0>(); c.step<1>(); c.step<2>(); c.step<3>(); c.step<4>(); c.step<5>(); c.step<6>();
    c.step<7>(); c.step<8>(); c.step<9>(); c.step<10>(); c.step<11>(); c.step<12>();
    c.tail();
    Batch q;
    float sc0;
    asm volatile("ds_read_b32 %0, %1 offset:64" : "=v"(sc0) : "v"(fbase) : "memory");
    batch_issue<3>(q, fbase, e0, e1);
    asm volatile("s_waitcnt lgkmcnt(0)" : "+v"(sc0) :: "memory");
    const float sc = batch_finish<3>(q, fbase, e0, e1, t0, sc0);
    batch_store<3>(q, ovoff, o0, o1, sc);
    asm volatile(".La2a_code_end:" ::: "memory");
    if (c.dead && lane == 0) __hip_atomic_store((unsigned*)(myws + WS_TMO), call + 1u, __ATOMIC_RELAXED, __HIP_MEMORY_SCOPE_AGENT);
  }
}

extern "C" void kernel_launch(int rank, int nranks, ncclComm_t comm, hipStream_t stream,
                              void* const* d_in, const int* in_sizes, int n_in,
                              void* d_out, int out_size, const int* in_dtypes, int out_dtype,
                              void* const* d_out_all, void* const* d_ws_all, size_t ws_bytes) {
  if (nranks != NRANKS || (unsigned)rank >= (unsigned)NRANKS || ws_bytes < WS_NEED || n_in != 4 ||
      in_sizes[0] != 4096 * 512 || in_sizes[1] != 4096 * 4096 || out_size != 512 * 4096) {
    fprintf(stderr, "unsupported configuration\n");
    return;
  }
  static bool init = false;
  if (!init) {
    HIP_CHECK(hipFuncSetAttribute((const void*)a2a_gemm_kernel, hipFuncAttributeMaxDynamicSharedMemorySize, LDS_BYTES));
    init = true;
  }
  Args a;
  a.x = (const uint8_t*)d_in[0]; a.W = (const uint8_t*)d_in[1];
  a.sx = (const float*)d_in[2]; a.sw = (const float*)d_in[3];
  a.out = (float*)d_out;
  a.myws = (uint8_t*)d_ws_all[rank];
  for (int p = 0; p < NRANKS; p++) a.peer[p] = (uint8_t*)d_ws_all[(rank + 1 + p) % NRANKS];
  a.rank = rank;
  a.slow = (int)((3u - (unsigned)((uintptr_t)d_in[1] >> 7)) & 7u);
  a2a_gemm_kernel<<<GRID, 512, LDS_BYTES, stream>>>(a);
}
```
